```python
import numpy as np

kernel_name = "gpu_gemm_a2a_m4096_k4096_n4096_f32_reluqe4m3"


ABSMAX_RTOL = 0.09

COMPUTE_DTYPE = "float32"
EPILOGUE = "relu_quant_e4m3"
IN_QUANT = None
OUT_QUANT = "e4m3"
IN_DTYPES = {"x": "float32", "w_mat": "float32"}
WORLD_SIZE = 8

def setup_inputs():

    x = np.random.randn(4096, 4096).astype(np.float32)
    w_mat = (np.random.randn(4096, 4096) / np.sqrt(4096)).astype(np.float32)
    return {"x": x, "w_mat": w_mat}

def _snap_fp8(x, mant_bits, max_abs, min_normal):
    a = np.abs(x.astype(np.float64))
    _m, _e = np.frexp(a)
    q = np.where(a >= min_normal, np.ldexp(1.0, _e - 1 - mant_bits),
                 min_normal * 2.0 ** -mant_bits)
    snapped = np.minimum(np.round(a / q) * q, max_abs)
    return (np.sign(x) * snapped).astype(np.float32)
def _quant_dequant(y, amax):
    scale = amax / 448.0
    q = _snap_fp8(y / scale, 3, 448.0, 2.0 ** -6)
    return (q * scale).astype(np.float32)
def _epilogue(y):
    y = np.maximum(y, 0.0)
    return _quant_dequant(y, np.abs(y).max())

def reference(x, w_mat):
    y = (x @ w_mat).astype(np.float32)
    return _epilogue(y)
```

```cpp
#include <hip/hip_runtime.h>
#include <rccl/rccl.h>
#include <cstdio>

#define HIP_CHECK(expr) do { hipError_t _e = (expr); if (_e != hipSuccess) { \
    fprintf(stderr, "HIP error %s at %s:%d\n", hipGetErrorString(_e), __FILE__, __LINE__); \
    return; } } while (0)

typedef unsigned u32x4 __attribute__((ext_vector_type(4)));
typedef unsigned u32x2 __attribute__((ext_vector_type(2)));
typedef float f32x4 __attribute__((ext_vector_type(4)));
typedef float f32x2 __attribute__((ext_vector_type(2)));
typedef __bf16 bf16x2 __attribute__((ext_vector_type(2)));
typedef __bf16 bf16x8 __attribute__((ext_vector_type(8)));
typedef unsigned long long u64;

#define CFG_BM 128
#define CFG_LEAD 2

constexpr int NR = 8;
constexpr int ML = 512, NN = 4096, KK = 4096;
constexpr int NSH = NN / NR;
constexpr int BM = CFG_BM, BN = 256;
constexpr int SPLIT = BM / 32;
constexpr int MT = ML / BM;
constexpr int NPX = (NN / BN) / (8 / SPLIT);
constexpr int KS = KK / SPLIT, STEPS = KS / 32;
constexpr int FI = BM / 16;
constexpr int XL = BM / 32;
constexpr int NWG = 256;
constexpr int XBUF = BM * 64;
constexpr int LDS_BYTES = 96 * 1024;

constexpr size_t WS_CNT = 0;
constexpr size_t WS_TMO = WS_CNT + NWG * 64;
constexpr size_t WS_DONE = WS_TMO + 256;
constexpr size_t WS_AMAX = WS_DONE + 1024;
constexpr size_t WS_XFLAG = WS_AMAX + 2 * NR * NWG * 8;
constexpr size_t WS_STAMP = WS_XFLAG + NWG * 4 * 64;
constexpr size_t WS_PART = 1 << 20;
constexpr size_t WS_END = WS_PART + (size_t)NWG * (SPLIT - 1) * 16384;
static_assert(WS_STAMP + NWG * 128 <= WS_PART, "layout");
constexpr u64 DEADLINE_TICKS = 50000000ull;
constexpr int AUX_AGENT = 16, AUX_SYS = 17;

struct Params {
    const float* x; const float* w;
    char* out[NR]; char* ws[NR];
    int rank; int pad;
};

__device__ __forceinline__ unsigned pk_bf16(float lo, float hi) {
    f32x2 v = {lo, hi}; bf16x2 b = __builtin_convertvector(v, bf16x2); return __builtin_bit_cast(unsigned, b);
}
__device__ __forceinline__ float asf(unsigned u) { return __builtin_bit_cast(float, u); }
__device__ __forceinline__ unsigned asu(float f) { return __builtin_bit_cast(unsigned, f); }
__device__ __forceinline__ float bf_lo(unsigned u) { return __builtin_bit_cast(float, u << 16); }
__device__ __forceinline__ float bf_hi(unsigned u) { return __builtin_bit_cast(float, u & 0xffff0000u); }
__device__ __forceinline__ __amdgpu_buffer_rsrc_t mk_rsrc(const void* p, unsigned bytes) {
    return __builtin_amdgcn_make_buffer_rsrc((void*)p, 0, (int)bytes, 0x00020000);
}
__device__ __forceinline__ u64 rt() { return __builtin_amdgcn_s_memrealtime(); }
#define STAMP(i) do {} while (0)

__device__ __forceinline__ int step_of(int t, int rho) {
    if (CFG_LEAD == 0) return t;
    constexpr int A = CFG_LEAD;
    if (A < MT) {
        int c = t + A;
        if (c < STEPS && (c % MT) == rho) return c;
        int d = (rho - t % MT + MT) % MT;
        int cn = t + d;
        if (d < A && cn < STEPS) {
            if (cn - A < 0) { return t == 0 ? cn : t - 1; }
            return t - 1;
        }
        return t;
    } else {
        if (t == 0) return rho;
        if (t - 1 >= STEPS - MT + rho) return t;
        return ((t - 1) % MT == rho) ? t - 1 + MT : t - 1;
    }
}

__global__ void __launch_bounds__(256, 1) fused_gemm_a2a(Params p) {
    extern __shared__ __attribute__((aligned(16))) char lds[];
    const int tid = threadIdx.x, lane = tid & 63;
    const int wave = __builtin_amdgcn_readfirstlane(tid >> 6);
    const int b = blockIdx.x, xcd = b & 7, j = b >> 3;
    const int kq = xcd % SPLIT, np = xcd / SPLIT, mi = j % MT, xr = j / MT, ni = np * NPX + xr;
    const int rho = (mi + xr) % MT;
    const int rank = p.rank;
    char* const ws = p.ws[rank];
    STAMP(0);

    unsigned call = 0;
    if (tid == 0) {
        call = __hip_atomic_fetch_add((unsigned*)(ws + WS_CNT + (size_t)b * 64), 1u, __ATOMIC_RELAXED, __HIP_MEMORY_SCOPE_AGENT) + 1u;
        *(unsigned*)(lds + 2 * XBUF) = call;
    }

    const __amdgpu_buffer_rsrc_t rx = mk_rsrc(p.x, ML * KK * 4), rw = mk_rsrc(p.w, KK * NN * 4);
    const int c16 = lane & 15, q = lane >> 4;
    const int c8 = tid & 7, xrow = tid >> 3;
    const unsigned xv = (unsigned)((mi * BM + xrow) * KK + kq * KS + c8 * 4) * 4u;
    const int wq = c8 >> 1;
    const int aq_w = (wq & 1) * 2 + (wq >> 1) * 12;
    const unsigned xw = (unsigned)((xrow >> 4) * 1024 + wq * 256 + (((xrow & 15) ^ aq_w) * 16) + (c8 & 1) * 8);
    const int aq_r = (q & 1) * 2 + (q >> 1) * 12;
    const unsigned xrd = (unsigned)(q * 256 + ((c16 ^ aq_r) * 16));
    const unsigned wv = (unsigned)((kq * KS + q * 8) * NN + ni * BN + wave * 64 + c16 * 4) * 4u;

    f32x4 acc[4][FI];
#pragma unroll
    for (int t = 0; t < 4; ++t)
#pragma unroll
        for (int i = 0; i < FI; ++i) acc[t][i] = f32x4{0.f, 0.f, 0.f, 0.f};

    u32x4 xr0[XL], xr1[XL], wr0[8], wr1[8];
    const int otab = step_of(lane < STEPS ? lane : STEPS - 1, rho);
    auto ldx = [&](u32x4* r, int t) {
        const int s = __builtin_amdgcn_readlane(otab, t);
#pragma unroll
        for (int i = 0; i < XL; ++i)
            r[i] = __builtin_amdgcn_raw_buffer_load_b128(rx, xv, s * 128 + ((i + kq) % XL) * (32 * KK * 4), 0);
    };
    auto ldw = [&](u32x4* r, int t) {
        const int s = __builtin_amdgcn_readlane(otab, t);
#pragma unroll
        for (int jj = 0; jj < 8; ++jj)
            r[jj] = __builtin_amdgcn_raw_buffer_load_b128(rw, wv, (s * 32 + jj) * (NN * 4), 0);
    };
    auto stx = [&](const u32x4* r, int buf) {
#pragma unroll
        for (int i = 0; i < XL; ++i) {
            f32x4 f = __builtin_bit_cast(f32x4, r[i]);
            u32x2 o = {pk_bf16(f[0], f[1]), pk_bf16(f[2], f[3])};
            *(u32x2*)(lds + buf * XBUF + i * 2048 + xw) = o;
        }
    };
    auto step = [&](u32x4* xnext, u32x4* wcur, int t, int buf, bool dost, bool dox, bool dow) {
        if (dost) stx(xnext, buf ^ 1);
        if (dox) ldx(xnext, t + 3);
        u32x4 wf[4];
#pragma unroll
        for (int tt = 0; tt < 4; ++tt)
#pragma unroll
            for (int h = 0; h < 4; ++h)
                wf[tt][h] = pk_bf16(asf(wcur[2 * h][tt]), asf(wcur[2 * h + 1][tt]));
        if (dow) ldw(wcur, t + 2);
#pragma unroll
        for (int i = 0; i < FI; ++i) {
            u32x4 xf = *(const u32x4*)(lds + buf * XBUF + i * 1024 + xrd);
#pragma unroll
            for (int tt = 0; tt < 4; ++tt)
                acc[tt][i] = __builtin_amdgcn_mfma_f32_16x16x32_bf16(__builtin_bit_cast(bf16x8, wf[tt]), __builtin_bit_cast(bf16x8, xf), acc[tt][i], 0, 0, 0);
        }
        __syncthreads();
    };

    ldx(xr0, 0); ldx(xr1, 1); ldw(wr0, 0);
    stx(xr0, 0);
    ldx(xr0, 2); ldw(wr1, 1);
    __syncthreads();
    for (int t = 0; t < STEPS - 4; t += 2) {
        step(xr1, wr0, t, 0, true, true, true);
        step(xr0, wr1, t + 1, 1, true, true, true);
    }
    step(xr1, wr0, STEPS - 4, 0, true, true, true);
    step(xr0, wr1, STEPS - 3, 1, true, false, true);
    step(xr1, wr0, STEPS - 2, 0, true, false, false);
    step(xr0, wr1, STEPS - 1, 1, false, false, false);
    call = *(const unsigned*)(lds + 2 * XBUF);
    call = __builtin_amdgcn_readfirstlane(call);
    STAMP(1);

    unsigned* const tmo = (unsigned*)(ws + WS_TMO);
    const __amdgpu_buffer_rsrc_t rws = mk_rsrc(ws, (unsigned)WS_END);
#pragma unroll
    for (int d = 1; d < SPLIT; ++d) {
        const int pb = (j << 3) | (np * SPLIT + (kq + d) % SPLIT);
        const unsigned base = (unsigned)(WS_PART + ((size_t)pb * (SPLIT - 1) + (d - 1)) * 16384) + wave * 4096;
#pragma unroll
        for (int ii = 0; ii < 2; ++ii)
#pragma unroll
            for (int h = 0; h < 2; ++h) {
                u32x4 o;
#pragma unroll
                for (int e = 0; e < 2; ++e) {
                    const int r = 2 * h + e;
                    o[2 * e] = pk_bf16(acc[0][2 * d + ii][r], acc[1][2 * d + ii][r]);
                    o[2 * e + 1] = pk_bf16(acc[2][2 * d + ii][r], acc[3][2 * d + ii][r]);
                }
                __builtin_amdgcn_raw_buffer_store_b128(o, rws, lane * 16 + (ii * 2 + h) * 1024, base, AUX_AGENT);
            }
    }
    asm volatile("s_waitcnt vmcnt(0)" ::: "memory");
    if (lane >= 1 && lane < SPLIT) {
        const int pb = (j << 3) | (np * SPLIT + (kq + lane) % SPLIT);
        __hip_atomic_store((unsigned*)(ws + WS_XFLAG + ((size_t)pb * 4 + wave) * 64) + (lane - 1), call, __ATOMIC_RELAXED, __HIP_MEMORY_SCOPE_AGENT);
    }
    STAMP(2);
    {
        const unsigned* fl = (const unsigned*)(ws + WS_XFLAG + ((size_t)b * 4 + wave) * 64) + (lane < SPLIT - 1 ? lane : 0);
        const u64 t0 = rt();
        while (true) {
            const unsigned v = __hip_atomic_load(fl, __ATOMIC_RELAXED, __HIP_MEMORY_SCOPE_AGENT);
            if (__all(v >= call)) break;
            if (__builtin_amdgcn_readfirstlane(__hip_atomic_load(tmo, __ATOMIC_RELAXED, __HIP_MEMORY_SCOPE_AGENT)) == call || rt() - t0 > DEADLINE_TICKS) {
                if (lane == 0) __hip_atomic_store(tmo, call, __ATOMIC_RELAXED, __HIP_MEMORY_SCOPE_AGENT);
                break;
            }
            __builtin_amdgcn_s_sleep(1);
        }
        __builtin_amdgcn_fence(__ATOMIC_ACQUIRE, "wavefront");
        asm volatile("" ::: "memory");
    }
    STAMP(3);
    float y[2][16];
    {
        u32x4 pr[SPLIT - 1][4];
#pragma unroll
        for (int d = 1; d < SPLIT; ++d) {
            const unsigned base = (unsigned)(WS_PART + ((size_t)b * (SPLIT - 1) + (d - 1)) * 16384) + wave * 4096;
#pragma unroll
            for (int u = 0; u < 4; ++u) pr[d - 1][u] = __builtin_amdgcn_raw_buffer_load_b128(rws, lane * 16 + u * 1024, base, AUX_AGENT);
        }
#pragma unroll
        for (int ii = 0; ii < 2; ++ii)
#pragma unroll
            for (int r = 0; r < 4; ++r)
#pragma unroll
                for (int tt = 0; tt < 4; ++tt) {
                    float s = acc[tt][ii][r];
#pragma unroll
                    for (int d = 1; d < SPLIT; ++d) {
                        const unsigned u = pr[d - 1][ii * 2 + (r >> 1)][(r & 1) * 2 + (tt >> 1)];
                        s += (tt & 1) ? bf_hi(u) : bf_lo(u);
                    }
                    y[ii][r * 4 + tt] = fmaxf(s, 0.f);
                }
    }
    float mx = 0.f;
#pragma unroll
    for (int ii = 0; ii < 2; ++ii)
#pragma unroll
        for (int e = 0; e < 16; ++e) mx = fmaxf(mx, y[ii][e]);
#pragma unroll
    for (int o = 32; o >= 1; o >>= 1) mx = fmaxf(mx, __shfl_xor(mx, o, 64));
    float* red = (float*)(lds + 2 * XBUF + 64);
    if (lane == 0) red[wave] = mx;
    __syncthreads();
    const unsigned par = call & 1u;
    if (tid < NR) {
        const float m4 = fmaxf(fmaxf(red[0], red[1]), fmaxf(red[2], red[3]));
        u64* g = (u64*)(p.ws[tid] + WS_AMAX) + (size_t)par * NR * NWG + rank * NWG + b;
        __hip_atomic_store(g, ((u64)call << 32) | asu(m4), __ATOMIC_RELAXED, __HIP_MEMORY_SCOPE_SYSTEM);
    }
    STAMP(4);
    {
        const unsigned base = (unsigned)(WS_AMAX + (size_t)par * NR * NWG * 8);
        const u64 t0 = rt();
        float gm;
        while (true) {
            u32x4 g[4];
#pragma unroll
            for (int u = 0; u < 4; ++u) g[u] = __builtin_amdgcn_raw_buffer_load_b128(rws, tid * 16 + u * 4096, base, AUX_SYS);
            bool ok = true; gm = 0.f;
#pragma unroll
            for (int u = 0; u < 4; ++u) {
                ok = ok && g[u][1] == call && g[u][3] == call;
                gm = fmaxf(gm, fmaxf(asf(g[u][0]), asf(g[u][2])));
            }
            if (__all(ok)) break;
            if (__builtin_amdgcn_readfirstlane(__hip_atomic_load(tmo, __ATOMIC_RELAXED, __HIP_MEMORY_SCOPE_AGENT)) == call || rt() - t0 > DEADLINE_TICKS) {
                if (lane == 0) __hip_atomic_store(tmo, call, __ATOMIC_RELAXED, __HIP_MEMORY_SCOPE_AGENT);
                break;
            }
            __builtin_amdgcn_s_sleep(2);
            asm volatile("" ::: "memory");
        }
#pragma unroll
        for (int o = 32; o >= 1; o >>= 1) gm = fmaxf(gm, __shfl_xor(gm, o, 64));
        if (lane == 0) red[4 + wave] = gm;
        asm volatile("s_waitcnt vmcnt(0)" ::: "memory");
        __syncthreads();
    }
    STAMP(5);
    const float amax = fmaxf(fmaxf(red[4], red[5]), fmaxf(red[6], red[7]));
    const float scale = amax / 448.f;
    const float inv = scale > 0.f ? 1.f / scale : 0.f;
    const bool bad = __hip_atomic_load(tmo, __ATOMIC_RELAXED, __HIP_MEMORY_SCOPE_AGENT) == call;
    const unsigned dst = (unsigned)(ni * BN) / NSH;
    if (dst < (unsigned)NR) {
        const __amdgpu_buffer_rsrc_t ro = mk_rsrc(p.out[dst], NN * NSH);
#pragma unroll
        for (int ii = 0; ii < 2; ++ii) {
            u32x4 o;
#pragma unroll
            for (int r = 0; r < 4; ++r) {
                int v = 0;
                v = __builtin_amdgcn_cvt_pk_fp8_f32(fminf(y[ii][4 * r] * inv, 448.f), fminf(y[ii][4 * r + 1] * inv, 448.f), v, false);
                v = __builtin_amdgcn_cvt_pk_fp8_f32(fminf(y[ii][4 * r + 2] * inv, 448.f), fminf(y[ii][4 * r + 3] * inv, 448.f), v, true);
                o[r] = bad ? 0x7f7f7f7fu : (unsigned)v;
            }
            const int row = rank * ML + mi * BM + kq * 32 + ii * 16 + c16;
            const int col = (ni * BN) % NSH + wave * 64 + q * 16;
            __builtin_amdgcn_raw_buffer_store_b128(o, ro, row * NSH + col, 0, AUX_SYS);
        }
        asm volatile("s_waitcnt vmcnt(0)" ::: "memory");
        __syncthreads();
        if (tid == 0) {
            const int idx = rank * 32 + ((ni & 1) * MT + mi) * SPLIT + kq;
            __hip_atomic_store((unsigned*)(p.ws[dst] + WS_DONE) + idx, call, __ATOMIC_RELAXED, __HIP_MEMORY_SCOPE_SYSTEM);
        }
    }
    STAMP(6);
    if (b == 0 && wave == 0) {
        const u64 t0 = rt();
        while (true) {
            const u32x4 g = __builtin_amdgcn_raw_buffer_load_b128(rws, lane * 16, (unsigned)WS_DONE, AUX_SYS);
            if (__all(g[0] >= call && g[1] >= call && g[2] >= call && g[3] >= call)) break;
            if (__builtin_amdgcn_readfirstlane(__hip_atomic_load(tmo, __ATOMIC_RELAXED, __HIP_MEMORY_SCOPE_AGENT)) == call || rt() - t0 > DEADLINE_TICKS) {
                if (lane == 0) {
                    __hip_atomic_store(tmo, call, __ATOMIC_RELAXED, __HIP_MEMORY_SCOPE_AGENT);
                    __hip_atomic_store((unsigned*)(p.out[rank] + (size_t)rank * ML * NSH), 0x7f7f7f7fu, __ATOMIC_RELAXED, __HIP_MEMORY_SCOPE_SYSTEM);
                }
                break;
            }
            __builtin_amdgcn_s_sleep(1);
            asm volatile("" ::: "memory");
        }
        STAMP(7);
    }
}

extern "C" void kernel_launch(int rank, int nranks, ncclComm_t comm,
                              hipStream_t stream,
                              void* const* d_in, const int* in_sizes, int n_in,
                              void* d_out, int out_size,
                              const int* in_dtypes, int out_dtype,
                              void* const* d_out_all,
                              void* const* d_ws_all, size_t ws_bytes) {
    if (nranks != NR || n_in != 2 || in_sizes[0] != ML * KK || in_sizes[1] != KK * NN || out_size != NN * NSH ||
        ws_bytes < WS_END || (unsigned)rank >= (unsigned)NR) {
        fprintf(stderr, "kernel_launch: unexpected problem shape\n");
        return;
    }
    static bool attr_set = false;
    if (!attr_set) {
        HIP_CHECK(hipFuncSetAttribute((const void*)fused_gemm_a2a, hipFuncAttributeMaxDynamicSharedMemorySize, LDS_BYTES));
        attr_set = true;
    }
    Params p;
    p.x = (const float*)d_in[0];
    p.w = (const float*)d_in[1];
    for (int r = 0; r < NR; ++r) { p.out[r] = (char*)d_out_all[r]; p.ws[r] = (char*)d_ws_all[r]; }
    p.rank = rank; p.pad = 0;
    fused_gemm_a2a<<<NWG, 256, LDS_BYTES, stream>>>(p);
    HIP_CHECK(hipGetLastError());
}
```

```cpp
#include <hip/hip_runtime.h>
#include <rccl/rccl.h>
#include <cstdio>

#define HIP_CHECK(expr) do { hipError_t _e = (expr); if (_e != hipSuccess) { \
    fprintf(stderr, "HIP error %s at %s:%d\n", hipGetErrorString(_e), __FILE__, __LINE__); \
    return; } } while (0)

typedef unsigned u32x4 __attribute__((ext_vector_type(4)));
typedef unsigned u32x2 __attribute__((ext_vector_type(2)));
typedef float f32x4 __attribute__((ext_vector_type(4)));
typedef float f32x32 __attribute__((ext_vector_type(32)));
typedef float f32x2 __attribute__((ext_vector_type(2)));
typedef __bf16 bf16x2 __attribute__((ext_vector_type(2)));
typedef unsigned long long u64;
#define GLBP __attribute__((address_space(1)))
#define LDSP __attribute__((address_space(3)))

constexpr unsigned PACE = 33;
constexpr unsigned READY_T = 60;
constexpr int CODEPF = 17;
constexpr unsigned long long ORD0 = 0xbc9a785634120ull, ORD1 = 0xcab8967452301ull;
constexpr unsigned long long ORDS0 = 0xb97ca85634120ull, ORDS1 = 0xca86b97452301ull;

constexpr int NR = 8;
constexpr int ML = 512, NN = 4096, KK = 4096;
constexpr int NSH = NN / NR;
constexpr int BM = 256, BN = 128, SPLIT = 4;
constexpr int KS = KK / SPLIT, NB = KS / 64;
constexpr int NF = 3;
constexpr int CV = 4;
constexpr int NWG = 256;
constexpr int SLOT = 49152, L_OWN = 2 * SLOT, L_CTL = 3 * SLOT, LDS_BYTES = L_CTL + 256;
constexpr int L_PK = SLOT, PK = 5120;

constexpr unsigned WS_CNT = 0;
constexpr unsigned WS_TMO = WS_CNT + NWG * 64;
constexpr unsigned WS_LMAX = WS_TMO + 256;
constexpr unsigned WS_GMAX = WS_LMAX + NWG * 8;
constexpr unsigned WS_PACE = WS_GMAX + 2 * 8 * 128;
constexpr unsigned WS_IFLAG = WS_PACE + NR * 128;
constexpr unsigned WS_XFLAG = WS_IFLAG + 8 * 512;
constexpr unsigned WS_IMG = 1u << 20;
constexpr unsigned IMG_GRP = (NB - NF) * 32768;
constexpr unsigned WS_PART = WS_IMG + 8 * IMG_GRP;
constexpr unsigned STG = 4352;
constexpr unsigned PRE = 2560;
constexpr unsigned WS_STAGE = WS_PART + NWG * (SPLIT - 1) * 16384;
constexpr unsigned WS_END = WS_STAGE + NWG * 4 * STG;
constexpr unsigned OOB = 0x40000000u;
static_assert(WS_XFLAG + NWG * 4 * 64 <= WS_IMG, "layout");
constexpr unsigned DEADLINE_TICKS = 50000000u;
constexpr int AUX_AGENT = 16, AUX_SYS = 17;

struct Params {
    const float* x; const float* w;
    char* my_out; char* my_ws;
    int rank; int pad;
    char* ws[NR];
};

__device__ __forceinline__ unsigned pk_bf16(float lo, float hi) {
    f32x2 v = {lo, hi}; bf16x2 b = __builtin_convertvector(v, bf16x2); return __builtin_bit_cast(unsigned, b);
}
__device__ __forceinline__ float asf(unsigned u) { return __builtin_bit_cast(float, u); }
__device__ __forceinline__ unsigned asu(float f) { return __builtin_bit_cast(unsigned, f); }
__device__ __forceinline__ float bf_lo(unsigned u) { return __builtin_bit_cast(float, u << 16); }
__device__ __forceinline__ float bf_hi(unsigned u) { return __builtin_bit_cast(float, u & 0xffff0000u); }
__device__ __forceinline__ __amdgpu_buffer_rsrc_t mk_rsrc(const void* p, unsigned bytes) {
    return __builtin_amdgcn_make_buffer_rsrc((void*)p, 0, (int)bytes, 0x00020000);
}
#define st128(v, r, voff, base, aux) __builtin_amdgcn_raw_buffer_store_b128(v, r, (voff) + (base), 0, aux)
__device__ __forceinline__ unsigned row_max(unsigned v) {
    v = max(v, (unsigned)__builtin_amdgcn_update_dpp(0, (int)v, 0xB1, 0xf, 0xf, true));
    v = max(v, (unsigned)__builtin_amdgcn_update_dpp(0, (int)v, 0x4E, 0xf, 0xf, true));
    v = max(v, (unsigned)__builtin_amdgcn_update_dpp(0, (int)v, 0x141, 0xf, 0xf, true));
    v = max(v, (unsigned)__builtin_amdgcn_update_dpp(0, (int)v, 0x140, 0xf, 0xf, true));
    return v;
}
__device__ __forceinline__ unsigned wave_max(unsigned v) {
    v = row_max(v);
    return max(max((unsigned)__builtin_amdgcn_readlane(v, 0), (unsigned)__builtin_amdgcn_readlane(v, 16)),
               max((unsigned)__builtin_amdgcn_readlane(v, 32), (unsigned)__builtin_amdgcn_readlane(v, 48)));
}
__device__ __forceinline__ unsigned wave_scan(unsigned v, unsigned& total) {
    unsigned s = v;
    s += (unsigned)__builtin_amdgcn_update_dpp(0, (int)s, 0x111, 0xf, 0xf, true);
    s += (unsigned)__builtin_amdgcn_update_dpp(0, (int)s, 0x112, 0xf, 0xf, true);
    s += (unsigned)__builtin_amdgcn_update_dpp(0, (int)s, 0x114, 0xf, 0xf, true);
    s += (unsigned)__builtin_amdgcn_update_dpp(0, (int)s, 0x118, 0xf, 0xf, true);
    s += (unsigned)__builtin_amdgcn_update_dpp(0, (int)s, 0x142, 0xa, 0xf, false);
    s += (unsigned)__builtin_amdgcn_update_dpp(0, (int)s, 0x143, 0xc, 0xf, false);
    total = __builtin_amdgcn_readlane(s, 63);
    return s - v;
}
__device__ __forceinline__ int lane_id() {
    int l; asm volatile("v_mbcnt_lo_u32_b32 %0, -1, 0\n\tv_mbcnt_hi_u32_b32 %0, -1, %0" : "=v"(l)); return l;
}
__device__ __forceinline__ u64 rt() { return __builtin_amdgcn_s_memrealtime(); }

__device__ __forceinline__ u32x4 quant16(const float* v, const unsigned* keep, float inv, unsigned nan) {
    u32x4 o;
#pragma unroll
    for (int r = 0; r < 4; ++r) {
        const f32x2 a = f32x2{v[r], v[4 + r]} * inv, c = f32x2{v[8 + r], v[12 + r]} * inv;
        int w = __builtin_amdgcn_cvt_pk_fp8_f32(a[0], a[1], 0, false);
        w = __builtin_amdgcn_cvt_pk_fp8_f32(c[0], c[1], w, true);
        o[r] = ((unsigned)w & keep[r]) | nan;
    }
    return o;
}

__global__ void __launch_bounds__(512, 2) fused_gemm_a2a(Params p) {
    extern __shared__ __attribute__((aligned(16))) char lds[];
    u64 pc0;
    asm volatile("s_getpc_b64 %0" : "=s"(pc0));
    const int wave8 = __builtin_amdgcn_readfirstlane(threadIdx.x >> 6), wave = wave8 & 3;
    const int b = blockIdx.x, xcd = b & 7, j = b >> 3;
    const int kq = xcd & 3, np = xcd >> 2, mi = j & 1, xr = j >> 1;
    const int ni = (xr >> 1) * 4 + (np ^ (1 & (xr >> 1))) * 2 + (xr & 1);
    const int rank = p.rank;
    char* const ws = p.my_ws;
    const unsigned ldsb = (unsigned)(size_t)lds;
    volatile LDSP unsigned* const ctl = (volatile LDSP unsigned*)(lds + L_CTL);
    const __amdgpu_buffer_rsrc_t rws = mk_rsrc(ws, WS_END);
    unsigned* const tmo = (unsigned*)(ws + WS_TMO);
    const int sidx = ((np * 2 + (xr & 1)) * 2 + mi) * 4 + kq;
    const unsigned sbase = WS_STAGE + (b * 4 + wave) * STG;

    if (__builtin_expect(wave8 >= 4, 1)) {
        char* dst_ws;
        unsigned call;
        {
            const int lane = threadIdx.x & 63, c16 = lane & 15, q = lane >> 4, c8 = lane & 7, l3 = lane >> 3;
            const __amdgpu_buffer_rsrc_t rx = mk_rsrc(p.x, ML * KK * 4), rw = mk_rsrc(p.w, KK * NN * 4);
            const int grp = kq * 2 + mi, prod = np * 16 + xr;
            const unsigned cv = (unsigned)(l3 * KK * 4 + c8 * 16);
            const unsigned cs = (unsigned)(l3 * 128 + ((c8 ^ l3) * 16));
            const int pw = ((ni + 16) & 31) * 4 + wave;
            unsigned co[CV];
#pragma unroll
            for (int c = 0; c < CV; ++c) {
                const int o = pw * CV + c, blk = o >> 5, oc = o & 31;
                const int row = mi * BM + (oc >> 4) * 128 + ((((oc >> 2) & 3) + kq) & 3) * 32 + (oc & 3) * 8;
                co[c] = blk < NB - NF ? (unsigned)(row * KK + kq * KS + (NF + blk) * 64) * 4u : OOB;
            }
            f32x32 cvr;
        asm volatile(
        "s_setprio 2\n\t"
        "s_nop 3\n\t"
        "buffer_load_dwordx4 v[136:139], %[cv], %[rx], %[co0] offen nt\n\t"
        "buffer_load_dwordx4 v[140:143], %[cv], %[rx], %[co0] offen offset:128 nt\n\t"
        "buffer_load_dwordx4 v[144:147], %[cv], %[rx], %[co1] offen nt\n\t"
        "buffer_load_dwordx4 v[148:151], %[cv], %[rx], %[co1] offen offset:128 nt\n\t"
        "buffer_load_dwordx4 v[152:155], %[cv], %[rx], %[co2] offen nt\n\t"
        "buffer_load_dwordx4 v[156:159], %[cv], %[rx], %[co2] offen offset:128 nt\n\t"
        "buffer_load_dwordx4 v[160:163], %[cv], %[rx], %[co3] offen nt\n\t"
        "buffer_load_dwordx4 v[164:167], %[cv], %[rx], %[co3] offen offset:128 nt\n\t"
        : "=&{v[136:167]}"(cvr)
        : [cv] "v"(cv), [rx] "s"(rx), [co0] "s"(co[0]), [co1] "s"(co[1]), [co2] "s"(co[2]), [co3] "s"(co[3])
        : "memory");

            __builtin_amdgcn_sched_barrier(0);
            int di = (ni >> 2) & (NR - 1);
            asm volatile("" : "+s"(di));
            dst_ws = p.ws[di];
            const unsigned cb = (pw * CV + CV <= (NB - NF) * 32) ? WS_IMG + grp * IMG_GRP + pw * CV * 1024 : OOB;
            static_assert(((NB - NF) * 32) % CV == 0, "a wave's pieces are all inside or all outside the image");
            const unsigned xv = cv + (unsigned)((mi * BM + (wave >> 1) * 128) * KK + kq * KS) * 4u;
            const int g0 = (wave & 1) * 2 + kq;
            const unsigned xw = ldsb + wave * 8192 + cs;
            const unsigned wv = (unsigned)((kq * KS + (wave >> 1) * 16 + q * 4) * NN + ni * BN + (wave & 1) * 64 + c16 * 4) * 4u;
            const unsigned wl = ldsb + 32768 + wave * 4096 + lane * 16;
            const unsigned dv = WS_IMG + grp * IMG_GRP + wave * 8192 + lane * 16;
            const unsigned db = ldsb + wave * 8192;
            const unsigned fo = lane == 0 ? WS_IFLAG + grp * 512 + (prod * 4 + wave) * 4 : OOB;
            const unsigned po = WS_IFLAG + grp * 512 + lane * 8;
            const unsigned ct = WS_CNT + b * 64 + wave8 * 4, tm = WS_TMO;
            const unsigned dl = DEADLINE_TICKS;
            static_assert(PRE > 2048 && PRE <= 3072, "three stores");
            const unsigned pv = sbase + lane * 16, pv2 = 2048 + lane * 16 < PRE ? pv + 2048 : OOB, mf = ldsb + L_CTL + (12 + wave) * 4;
            static_assert(NF == 3 && NB == 16, "order tables");
            const int c1 = ((mi ? 0x201 : 0x120) >> (4 * (lane & 3))) & 3;
            const int c2 = (int)(((((xr & 1) && ni != 15) ? (mi ? ORD1 : ORD0) : (mi ? ORDS1 : ORDS0)) >> (4 * ((lane - NF) & 15))) & 15);
            const int blk = lane < NF ? (c1 + xr) % NF : NF + (c2 + 2 * xr + 6 * kq) % (NB - NF);
            const unsigned otx = lane < NF ? blk * 256u : (lane < NB ? (blk - NF) * 32768u : OOB);
            const unsigned otw = lane < NB ? (unsigned)blk << 20 : OOB;
        asm volatile(
        "s_nop 3\n\t"
        "v_mov_b32 v174, 1\n\t"
        "s_mov_b64 exec, 1\n\t"
        "buffer_atomic_add v174, off, %[rs], %[ct] sc0\n\t"
        "s_mov_b64 exec, -1\n\t"
        "s_mov_b32 s66, m0\n\t"
        "s_memrealtime s[64:65]\n\t"
        "s_add_u32 s40, %[g0], 0\n\t"
        "s_and_b32 s40, s40, 3\n\t"
        "s_lshl_b32 s40, s40, 19\n\t"
        "s_add_u32 s41, s40, 131072\n\t"
        "s_add_u32 s42, s41, 131072\n\t"
        "s_add_u32 s43, s42, 131072\n\t"
        "s_add_u32 s44, %[g0], 1\n\t"
        "s_and_b32 s44, s44, 3\n\t"
        "s_lshl_b32 s44, s44, 19\n\t"
        "s_add_u32 s45, s44, 131072\n\t"
        "s_add_u32 s46, s45, 131072\n\t"
        "s_add_u32 s47, s46, 131072\n\t"
        "s_mov_b32 s48, 16384\n\t"
        "s_mov_b32 s49, 32768\n\t"
        "s_mov_b32 s50, 49152\n\t"
        "s_mov_b32 s51, 524288\n\t"
        "s_mov_b32 s52, 540672\n\t"
        "s_mov_b32 s53, 557056\n\t"
        "s_mov_b32 s54, 573440\n\t"
        "s_mov_b32 s55, 4096\n\t"
        "v_readlane_b32 s57, %[otx], 0\n\t"
        "v_readlane_b32 s58, %[otw], 0\n\t"
        "s_nop 1\n\t"
        "v_add_u32 v170, s57, %[xv]\n\t"
        "v_add_u32 v171, s58, %[wv]\n\t"
        "s_waitcnt lgkmcnt(0)\n\t"
        "s_mov_b32 s67, s64\n\t"
        "buffer_load_dwordx4 v[24:27], v170, %[rx], s40 offen\n\t"
        "buffer_load_dwordx4 v[28:31], v170, %[rx], s40 offen offset:128\n\t"
        "s_waitcnt vmcnt(9)\n\t"
        "v_cvt_pk_bf16_f32 v136, v136, v137\n\t"
        "v_cvt_pk_bf16_f32 v137, v138, v139\n\t"
        "v_cvt_pk_bf16_f32 v138, v140, v141\n\t"
        "v_cvt_pk_bf16_f32 v139, v142, v143\n\t"
        "buffer_store_dwordx4 v[136:139], %[cs], %[rs], %[cb] offen offset:0 sc0 sc1\n\t"
        "buffer_load_dwordx4 v[32:35], v170, %[rx], s41 offen\n\t"
        "buffer_load_dwordx4 v[36:39], v170, %[rx], s41 offen offset:128\n\t"
        "s_waitcnt vmcnt(10)\n\t"
        "v_cvt_pk_bf16_f32 v144, v144, v145\n\t"
        "v_cvt_pk_bf16_f32 v145, v146, v147\n\t"
        "v_cvt_pk_bf16_f32 v146, v148, v149\n\t"
        "v_cvt_pk_bf16_f32 v147, v150, v151\n\t"
        "buffer_store_dwordx4 v[144:147], %[cs], %[rs], %[cb] offen offset:1024 sc0 sc1\n\t"
        "buffer_load_dwordx4 v[40:43], v170, %[rx], s42 offen\n\t"
        "buffer_load_dwordx4 v[44:47], v170, %[rx], s42 offen offset:128\n\t"
        "s_waitcnt vmcnt(11)\n\t"
        "v_cvt_pk_bf16_f32 v152, v152, v153\n\t"
        "v_cvt_pk_bf16_f32 v153, v154, v155\n\t"
        "v_cvt_pk_bf16_f32 v154, v156, v157\n\t"
        "v_cvt_pk_bf16_f32 v155, v158, v159\n\t"
        "buffer_store_dwordx4 v[152:155], %[cs], %[rs], %[cb] offen offset:2048 sc0 sc1\n\t"
        "buffer_load_dwordx4 v[48:51], v170, %[rx], s43 offen\n\t"
        "buffer_load_dwordx4 v[52:55], v170, %[rx], s43 offen offset:128\n\t"
        "s_waitcnt vmcnt(12)\n\t"
        "v_cvt_pk_bf16_f32 v160, v160, v161\n\t"
        "v_cvt_pk_bf16_f32 v161, v162, v163\n\t"
        "v_cvt_pk_bf16_f32 v162, v164, v165\n\t"
        "v_cvt_pk_bf16_f32 v163, v166, v167\n\t"
        "buffer_store_dwordx4 v[160:163], %[cs], %[rs], %[cb] offen offset:3072 sc0 sc1\n\t"
        "buffer_load_dwordx4 v[56:59], v170, %[rx], s44 offen\n\t"
        "buffer_load_dwordx4 v[60:63], v170, %[rx], s44 offen offset:128\n\t"
        "buffer_load_dwordx4 v[64:67], v170, %[rx], s45 offen\n\t"
        "buffer_load_dwordx4 v[68:71], v170, %[rx], s45 offen offset:128\n\t"
        "buffer_load_dwordx4 v[72:75], v170, %[rx], s46 offen\n\t"
        "buffer_load_dwordx4 v[76:79], v170, %[rx], s46 offen offset:128\n\t"
        "buffer_load_dwordx4 v[80:83], v170, %[rx], s47 offen\n\t"
        "buffer_load_dwordx4 v[84:87], v170, %[rx], s47 offen offset:128\n\t"
        "buffer_load_dwordx4 v[88:91], v171, %[rw], 0 offen\n\t"
        "buffer_load_dwordx4 v[92:95], v171, %[rw], s48 offen\n\t"
        "buffer_load_dwordx4 v[96:99], v171, %[rw], s49 offen\n\t"
        "buffer_load_dwordx4 v[100:103], v171, %[rw], s50 offen\n\t"
        "buffer_load_dwordx4 v[104:107], v171, %[rw], s51 offen\n\t"
        "buffer_load_dwordx4 v[108:111], v171, %[rw], s52 offen\n\t"
        "buffer_load_dwordx4 v[112:115], v171, %[rw], s53 offen\n\t"
        "buffer_load_dwordx4 v[116:119], v171, %[rw], s54 offen\n\t"
        "s_waitcnt vmcnt(16)\n\t"
        "v_readfirstlane_b32 %[call], v174\n\t"
        "s_nop 3\n\t"
        "s_add_u32 %[call], %[call], 1\n\t"
        "v_mov_b32 v174, %[call]\n\t"
        "buffer_store_dword v174, %[fo], %[rs], 0 offen sc0 sc1\n\t"
        "v_readlane_b32 s57, %[otx], 1\n\t"
        "v_readlane_b32 s58, %[otw], 1\n\t"
        "s_nop 1\n\t"
        "v_add_u32 v170, s57, %[xv]\n\t"
        "v_add_u32 v171, s58, %[wv]\n\t"
        "v_add_u32 v172, 0, %[xw]\n\t"
        "v_add_u32 v173, 0, %[wl]\n\t"
        "v_cvt_pk_bf16_f32 v24, v24, v25\n\t"
        "v_cvt_pk_bf16_f32 v25, v26, v27\n\t"
        "v_cvt_pk_bf16_f32 v26, v28, v29\n\t"
        "v_cvt_pk_bf16_f32 v27, v30, v31\n\t"
        "ds_write_b128 v172, v[24:27] offset:0\n\t"
        "buffer_load_dwordx4 v[24:27], v170, %[rx], s40 offen\n\t"
        "buffer_load_dwordx4 v[28:31], v170, %[rx], s40 offen offset:128\n\t"
        "v_cvt_pk_bf16_f32 v32, v32, v33\n\t"
        "v_cvt_pk_bf16_f32 v33, v34, v35\n\t"
        "v_cvt_pk_bf16_f32 v34, v36, v37\n\t"
        "v_cvt_pk_bf16_f32 v35, v38, v39\n\t"
        "ds_write_b128 v172, v[32:35] offset:1024\n\t"
        "buffer_load_dwordx4 v[32:35], v170, %[rx], s41 offen\n\t"
        "buffer_load_dwordx4 v[36:39], v170, %[rx], s41 offen offset:128\n\t"
        "v_cvt_pk_bf16_f32 v40, v40, v41\n\t"
        "v_cvt_pk_bf16_f32 v41, v42, v43\n\t"
        "v_cvt_pk_bf16_f32 v42, v44, v45\n\t"
        "v_cvt_pk_bf16_f32 v43, v46, v47\n\t"
        "ds_write_b128 v172, v[40:43] offset:2048\n\t"
        "buffer_load_dwordx4 v[40:43], v170, %[rx], s42 offen\n\t"
        "buffer_load_dwordx4 v[44:47], v170, %[rx], s42 offen offset:128\n\t"
        "v_cvt_pk_bf16_f32 v48, v48, v49\n\t"
        "v_cvt_pk_bf16_f32 v49, v50, v51\n\t"
        "v_cvt_pk_bf16_f32 v50, v52, v53\n\t"
        "v_cvt_pk_bf16_f32 v51, v54, v55\n\t"
        "ds_write_b128 v172, v[48:51] offset:3072\n\t"
        "buffer_load_dwordx4 v[48:51], v170, %[rx], s43 offen\n\t"
        "buffer_load_dwordx4 v[52:55], v170, %[rx], s43 offen offset:128\n\t"
        "s_waitcnt vmcnt(23)\n\t"
        "v_cvt_pk_bf16_f32 v56, v56, v57\n\t"
        "v_cvt_pk_bf16_f32 v57, v58, v59\n\t"
        "v_cvt_pk_bf16_f32 v58, v60, v61\n\t"
        "v_cvt_pk_bf16_f32 v59, v62, v63\n\t"
        "ds_write_b128 v172, v[56:59] offset:4096\n\t"
        "buffer_load_dwordx4 v[56:59], v170, %[rx], s44 offen\n\t"
        "buffer_load_dwordx4 v[60:63], v170, %[rx], s44 offen offset:128\n\t"
        "s_waitcnt vmcnt(23)\n\t"
        "v_cvt_pk_bf16_f32 v64, v64, v65\n\t"
        "v_cvt_pk_bf16_f32 v65, v66, v67\n\t"
        "v_cvt_pk_bf16_f32 v66, v68, v69\n\t"
        "v_cvt_pk_bf16_f32 v67, v70, v71\n\t"
        "ds_write_b128 v172, v[64:67] offset:5120\n\t"
        "buffer_load_dwordx4 v[64:67], v170, %[rx], s45 offen\n\t"
        "buffer_load_dwordx4 v[68:71], v170, %[rx], s45 offen offset:128\n\t"
        "s_waitcnt vmcnt(23)\n\t"
        "v_cvt_pk_bf16_f32 v72, v72, v73\n\t"
        "v_cvt_pk_bf16_f32 v73, v74, v75\n\t"
        "v_cvt_pk_bf16_f32 v74, v76, v77\n\t"
        "v_cvt_pk_bf16_f32 v75, v78, v79\n\t"
        "ds_write_b128 v172, v[72:75] offset:6144\n\t"
        "buffer_load_dwordx4 v[72:75], v170, %[rx], s46 offen\n\t"
        "buffer_load_dwordx4 v[76:79], v170, %[rx], s46 offen offset:128\n\t"
        "s_waitcnt vmcnt(23)\n\t"
        "v_cvt_pk_bf16_f32 v80, v80, v81\n\t"
        "v_cvt_pk_bf16_f32 v81, v82, v83\n\t"
        "v_cvt_pk_bf16_f32 v82, v84, v85\n\t"
        "v_cvt_pk_bf16_f32 v83, v86, v87\n\t"
        "ds_write_b128 v172, v[80:83] offset:7168\n\t"
        "buffer_load_dwordx4 v[80:83], v170, %[rx], s47 offen\n\t"
        "buffer_load_dwordx4 v[84:87], v170, %[rx], s47 offen offset:128\n\t"
        "s_waitcnt vmcnt(23)\n\t"
        "v_cvt_pk_bf16_f32 v120, v88, v92\n\t"
        "v_cvt_pk_bf16_f32 v124, v89, v93\n\t"
        "v_cvt_pk_bf16_f32 v128, v90, v94\n\t"
        "v_cvt_pk_bf16_f32 v132, v91, v95\n\t"
        "buffer_load_dwordx4 v[88:91], v171, %[rw], 0 offen\n\t"
        "buffer_load_dwordx4 v[92:95], v171, %[rw], s48 offen\n\t"
        "s_waitcnt vmcnt(23)\n\t"
        "v_cvt_pk_bf16_f32 v121, v96, v100\n\t"
        "v_cvt_pk_bf16_f32 v125, v97, v101\n\t"
        "v_cvt_pk_bf16_f32 v129, v98, v102\n\t"
        "v_cvt_pk_bf16_f32 v133, v99, v103\n\t"
        "buffer_load_dwordx4 v[96:99], v171, %[rw], s49 offen\n\t"
        "buffer_load_dwordx4 v[100:103], v171, %[rw], s50 offen\n\t"
        "s_waitcnt vmcnt(23)\n\t"
        "v_cvt_pk_bf16_f32 v122, v104, v108\n\t"
        "v_cvt_pk_bf16_f32 v126, v105, v109\n\t"
        "v_cvt_pk_bf16_f32 v130, v106, v110\n\t"
        "v_cvt_pk_bf16_f32 v134, v107, v111\n\t"
        "buffer_load_dwordx4 v[104:107], v171, %[rw], s51 offen\n\t"
        "buffer_load_dwordx4 v[108:111], v171, %[rw], s52 offen\n\t"
        "s_waitcnt vmcnt(23)\n\t"
        "v_cvt_pk_bf16_f32 v123, v112, v116\n\t"
        "v_cvt_pk_bf16_f32 v127, v113, v117\n\t"
        "v_cvt_pk_bf16_f32 v131, v114, v118\n\t"
        "v_cvt_pk_bf16_f32 v135, v115, v119\n\t"
        "ds_write_b128 v173, v[120:123] offset:0\n\t"
        "ds_write_b128 v173, v[124:127] offset:1024\n\t"
        "ds_write_b128 v173, v[128:131] offset:2048\n\t"
        "ds_write_b128 v173, v[132:135] offset:3072\n\t"
        "buffer_load_dwordx4 v[112:115], v171, %[rw], s53 offen\n\t"
        "buffer_load_dwordx4 v[116:119], v171, %[rw], s54 offen\n\t"
        "s_waitcnt lgkmcnt(0)\n\t"
        "s_barrier\n\t"
        "v_readlane_b32 s57, %[otx], 2\n\t"
        "v_readlane_b32 s58, %[otw], 2\n\t"
        "s_nop 1\n\t"
        "v_add_u32 v170, s57, %[xv]\n\t"
        "v_add_u32 v171, s58, %[wv]\n\t"
        "v_add_u32 v172, 49152, %[xw]\n\t"
        "v_add_u32 v173, 49152, %[wl]\n\t"
        "s_waitcnt vmcnt(22)\n\t"
        "v_cvt_pk_bf16_f32 v24, v24, v25\n\t"
        "v_cvt_pk_bf16_f32 v25, v26, v27\n\t"
        "v_cvt_pk_bf16_f32 v26, v28, v29\n\t"
        "v_cvt_pk_bf16_f32 v27, v30, v31\n\t"
        "ds_write_b128 v172, v[24:27] offset:0\n\t"
        "buffer_load_dwordx4 v[24:27], v170, %[rx], s40 offen\n\t"
        "buffer_load_dwordx4 v[28:31], v170, %[rx], s40 offen offset:128\n\t"
        "s_waitcnt vmcnt(22)\n\t"
        "v_cvt_pk_bf16_f32 v32, v32, v33\n\t"
        "v_cvt_pk_bf16_f32 v33, v34, v35\n\t"
        "v_cvt_pk_bf16_f32 v34, v36, v37\n\t"
        "v_cvt_pk_bf16_f32 v35, v38, v39\n\t"
        "ds_write_b128 v172, v[32:35] offset:1024\n\t"
        "buffer_load_dwordx4 v[32:35], v170, %[rx], s41 offen\n\t"
        "buffer_load_dwordx4 v[36:39], v170, %[rx], s41 offen offset:128\n\t"
        "s_waitcnt vmcnt(22)\n\t"
        "v_cvt_pk_bf16_f32 v40, v40, v41\n\t"
        "v_cvt_pk_bf16_f32 v41, v42, v43\n\t"
        "v_cvt_pk_bf16_f32 v42, v44, v45\n\t"
        "v_cvt_pk_bf16_f32 v43, v46, v47\n\t"
        "ds_write_b128 v172, v[40:43] offset:2048\n\t"
        "buffer_load_dwordx4 v[40:43], v170, %[rx], s42 offen\n\t"
        "buffer_load_dwordx4 v[44:47], v170, %[rx], s42 offen offset:128\n\t"
        "s_waitcnt vmcnt(22)\n\t"
        "v_cvt_pk_bf16_f32 v48, v48, v49\n\t"
        "v_cvt_pk_bf16_f32 v49, v50, v51\n\t"
        "v_cvt_pk_bf16_f32 v50, v52, v53\n\t"
        "v_cvt_pk_bf16_f32 v51, v54, v55\n\t"
        "ds_write_b128 v172, v[48:51] offset:3072\n\t"
        "buffer_load_dwordx4 v[48:51], v170, %[rx], s43 offen\n\t"
        "buffer_load_dwordx4 v[52:55], v170, %[rx], s43 offen offset:128\n\t"
        "s_waitcnt vmcnt(22)\n\t"
        "v_cvt_pk_bf16_f32 v56, v56, v57\n\t"
        "v_cvt_pk_bf16_f32 v57, v58, v59\n\t"
        "v_cvt_pk_bf16_f32 v58, v60, v61\n\t"
        "v_cvt_pk_bf16_f32 v59, v62, v63\n\t"
        "ds_write_b128 v172, v[56:59] offset:4096\n\t"
        "buffer_load_dwordx4 v[56:59], v170, %[rx], s44 offen\n\t"
        "buffer_load_dwordx4 v[60:63], v170, %[rx], s44 offen offset:128\n\t"
        "s_waitcnt vmcnt(22)\n\t"
        "v_cvt_pk_bf16_f32 v64, v64, v65\n\t"
        "v_cvt_pk_bf16_f32 v65, v66, v67\n\t"
        "v_cvt_pk_bf16_f32 v66, v68, v69\n\t"
        "v_cvt_pk_bf16_f32 v67, v70, v71\n\t"
        "ds_write_b128 v172, v[64:67] offset:5120\n\t"
        "buffer_load_dwordx4 v[64:67], v170, %[rx], s45 offen\n\t"
        "buffer_load_dwordx4 v[68:71], v170, %[rx], s45 offen offset:128\n\t"
        "s_waitcnt vmcnt(22)\n\t"
        "v_cvt_pk_bf16_f32 v72, v72, v73\n\t"
        "v_cvt_pk_bf16_f32 v73, v74, v75\n\t"
        "v_cvt_pk_bf16_f32 v74, v76, v77\n\t"
        "v_cvt_pk_bf16_f32 v75, v78, v79\n\t"
        "ds_write_b128 v172, v[72:75] offset:6144\n\t"
        "buffer_load_dwordx4 v[72:75], v170, %[rx], s46 offen\n\t"
        "buffer_load_dwordx4 v[76:79], v170, %[rx], s46 offen offset:128\n\t"
        "s_waitcnt vmcnt(22)\n\t"
        "v_cvt_pk_bf16_f32 v80, v80, v81\n\t"
        "v_cvt_pk_bf16_f32 v81, v82, v83\n\t"
        "v_cvt_pk_bf16_f32 v82, v84, v85\n\t"
        "v_cvt_pk_bf16_f32 v83, v86, v87\n\t"
        "ds_write_b128 v172, v[80:83] offset:7168\n\t"
        "buffer_load_dwordx4 v[80:83], v170, %[rx], s47 offen\n\t"
        "buffer_load_dwordx4 v[84:87], v170, %[rx], s47 offen offset:128\n\t"
        "buffer_load_dwordx2 v[168:169], %[po], %[rs], 0 offen sc0 sc1\n\t"
        "s_waitcnt vmcnt(23)\n\t"
        "v_cvt_pk_bf16_f32 v120, v88, v92\n\t"
        "v_cvt_pk_bf16_f32 v124, v89, v93\n\t"
        "v_cvt_pk_bf16_f32 v128, v90, v94\n\t"
        "v_cvt_pk_bf16_f32 v132, v91, v95\n\t"
        "buffer_load_dwordx4 v[88:91], v171, %[rw], 0 offen\n\t"
        "buffer_load_dwordx4 v[92:95], v171, %[rw], s48 offen\n\t"
        "s_waitcnt vmcnt(23)\n\t"
        "v_cvt_pk_bf16_f32 v121, v96, v100\n\t"
        "v_cvt_pk_bf16_f32 v125, v97, v101\n\t"
        "v_cvt_pk_bf16_f32 v129, v98, v102\n\t"
        "v_cvt_pk_bf16_f32 v133, v99, v103\n\t"
        "buffer_load_dwordx4 v[96:99], v171, %[rw], s49 offen\n\t"
        "buffer_load_dwordx4 v[100:103], v171, %[rw], s50 offen\n\t"
        "s_waitcnt vmcnt(23)\n\t"
        "v_cvt_pk_bf16_f32 v122, v104, v108\n\t"
        "v_cvt_pk_bf16_f32 v126, v105, v109\n\t"
        "v_cvt_pk_bf16_f32 v130, v106, v110\n\t"
        "v_cvt_pk_bf16_f32 v134, v107, v111\n\t"
        "buffer_load_dwordx4 v[104:107], v171, %[rw], s51 offen\n\t"
        "buffer_load_dwordx4 v[108:111], v171, %[rw], s52 offen\n\t"
        "s_waitcnt vmcnt(23)\n\t"
        "v_cvt_pk_bf16_f32 v123, v112, v116\n\t"
        "v_cvt_pk_bf16_f32 v127, v113, v117\n\t"
        "v_cvt_pk_bf16_f32 v131, v114, v118\n\t"
        "v_cvt_pk_bf16_f32 v135, v115, v119\n\t"
        "ds_write_b128 v173, v[120:123] offset:0\n\t"
        "ds_write_b128 v173, v[124:127] offset:1024\n\t"
        "ds_write_b128 v173, v[128:131] offset:2048\n\t"
        "ds_write_b128 v173, v[132:135] offset:3072\n\t"
        "buffer_load_dwordx4 v[112:115], v171, %[rw], s53 offen\n\t"
        "buffer_load_dwordx4 v[116:119], v171, %[rw], s54 offen\n\t"
        "s_waitcnt lgkmcnt(0)\n\t"
        "s_barrier\n\t"
        "v_readlane_b32 s57, %[otx], 3\n\t"
        "v_readlane_b32 s58, %[otw], 3\n\t"
        "s_nop 1\n\t"
        "v_add_u32 v170, s57, %[dv]\n\t"
        "v_add_u32 v171, s58, %[wv]\n\t"
        "v_add_u32 v172, 98304, %[xw]\n\t"
        "v_add_u32 v173, 98304, %[wl]\n\t"
        "s_waitcnt vmcnt(23)\n\t"
        "v_cvt_pk_bf16_f32 v24, v24, v25\n\t"
        "v_cvt_pk_bf16_f32 v25, v26, v27\n\t"
        "v_cvt_pk_bf16_f32 v26, v28, v29\n\t"
        "v_cvt_pk_bf16_f32 v27, v30, v31\n\t"
        "ds_write_b128 v172, v[24:27] offset:0\n\t"
        "s_waitcnt vmcnt(21)\n\t"
        "v_cvt_pk_bf16_f32 v32, v32, v33\n\t"
        "v_cvt_pk_bf16_f32 v33, v34, v35\n\t"
        "v_cvt_pk_bf16_f32 v34, v36, v37\n\t"
        "v_cvt_pk_bf16_f32 v35, v38, v39\n\t"
        "ds_write_b128 v172, v[32:35] offset:1024\n\t"
        "s_waitcnt vmcnt(19)\n\t"
        "v_cvt_pk_bf16_f32 v40, v40, v41\n\t"
        "v_cvt_pk_bf16_f32 v41, v42, v43\n\t"
        "v_cvt_pk_bf16_f32 v42, v44, v45\n\t"
        "v_cvt_pk_bf16_f32 v43, v46, v47\n\t"
        "ds_write_b128 v172, v[40:43] offset:2048\n\t"
        "s_waitcnt vmcnt(17)\n\t"
        "v_cvt_pk_bf16_f32 v48, v48, v49\n\t"
        "v_cvt_pk_bf16_f32 v49, v50, v51\n\t"
        "v_cvt_pk_bf16_f32 v50, v52, v53\n\t"
        "v_cvt_pk_bf16_f32 v51, v54, v55\n\t"
        "ds_write_b128 v172, v[48:51] offset:3072\n\t"
        "s_waitcnt vmcnt(15)\n\t"
        "v_cvt_pk_bf16_f32 v56, v56, v57\n\t"
        "v_cvt_pk_bf16_f32 v57, v58, v59\n\t"
        "v_cvt_pk_bf16_f32 v58, v60, v61\n\t"
        "v_cvt_pk_bf16_f32 v59, v62, v63\n\t"
        "ds_write_b128 v172, v[56:59] offset:4096\n\t"
        "s_waitcnt vmcnt(13)\n\t"
        "v_cvt_pk_bf16_f32 v64, v64, v65\n\t"
        "v_cvt_pk_bf16_f32 v65, v66, v67\n\t"
        "v_cvt_pk_bf16_f32 v66, v68, v69\n\t"
        "v_cvt_pk_bf16_f32 v67, v70, v71\n\t"
        "ds_write_b128 v172, v[64:67] offset:5120\n\t"
        "s_waitcnt vmcnt(11)\n\t"
        "v_cvt_pk_bf16_f32 v72, v72, v73\n\t"
        "v_cvt_pk_bf16_f32 v73, v74, v75\n\t"
        "v_cvt_pk_bf16_f32 v74, v76, v77\n\t"
        "v_cvt_pk_bf16_f32 v75, v78, v79\n\t"
        "ds_write_b128 v172, v[72:75] offset:6144\n\t"
        "s_waitcnt vmcnt(9)\n\t"
        "v_cvt_pk_bf16_f32 v80, v80, v81\n\t"
        "v_cvt_pk_bf16_f32 v81, v82, v83\n\t"
        "v_cvt_pk_bf16_f32 v82, v84, v85\n\t"
        "v_cvt_pk_bf16_f32 v83, v86, v87\n\t"
        "ds_write_b128 v172, v[80:83] offset:7168\n\t"
        "s_waitcnt vmcnt(8)\n\t"
        "v_min_u32 v168, v168, v169\n\t"
        "v_cmp_ge_u32 vcc, v168, v174\n\t"
        "s_nop 3\n\t"
        "s_cmp_eq_u64 vcc, -1\n\t"
        "s_cbranch_scc1 XOK%=\n\t"
        "s_memrealtime s[62:63]\n\t"
        "XSPIN%=:\n\t"
        "buffer_load_dwordx2 v[168:169], %[po], %[rs], 0 offen sc0 sc1\n\t"
        "buffer_load_dword v175, off, %[rs], %[tm] sc1\n\t"
        "s_memrealtime s[64:65]\n\t"
        "s_waitcnt vmcnt(0) lgkmcnt(0)\n\t"
        "v_readfirstlane_b32 s57, v175\n\t"
        "v_min_u32 v168, v168, v169\n\t"
        "v_cmp_ge_u32 vcc, v168, v174\n\t"
        "s_nop 3\n\t"
        "s_cmp_eq_u64 vcc, -1\n\t"
        "s_cbranch_scc1 XOK%=\n\t"
        "s_cmp_eq_u32 s57, %[call]\n\t"
        "s_cbranch_scc1 XOK%=\n\t"
        "s_sleep 4\n\t"
        "s_sub_u32 s64, s64, s62\n\t"
        "s_cmp_lt_u32 s64, %[dl]\n\t"
        "s_cbranch_scc1 XSPIN%=\n\t"
        "buffer_store_dword v174, off, %[rs], %[tm] sc1\n\t"
        "XOK%=:\n\t"
        "s_add_u32 s60, %[db], 0\n\t"
        "s_mov_b32 m0, s60\n\t"
        "s_nop 0\n\t"
        "buffer_load_dwordx4 v170, %[rs], 0 offen offset:0 sc0 sc1 lds\n\t"
        "buffer_load_dwordx4 v170, %[rs], 0 offen offset:1024 sc0 sc1 lds\n\t"
        "buffer_load_dwordx4 v170, %[rs], 0 offen offset:2048 sc0 sc1 lds\n\t"
        "buffer_load_dwordx4 v170, %[rs], 0 offen offset:3072 sc0 sc1 lds\n\t"
        "s_add_u32 m0, s60, 4096\n\t"
        "s_nop 0\n\t"
        "buffer_load_dwordx4 v170, %[rs], s55 offen offset:0 sc0 sc1 lds\n\t"
        "buffer_load_dwordx4 v170, %[rs], s55 offen offset:1024 sc0 sc1 lds\n\t"
        "buffer_load_dwordx4 v170, %[rs], s55 offen offset:2048 sc0 sc1 lds\n\t"
        "buffer_load_dwordx4 v170, %[rs], s55 offen offset:3072 sc0 sc1 lds\n\t"
        "s_waitcnt vmcnt(14)\n\t"
        "v_cvt_pk_bf16_f32 v120, v88, v92\n\t"
        "v_cvt_pk_bf16_f32 v124, v89, v93\n\t"
        "v_cvt_pk_bf16_f32 v128, v90, v94\n\t"
        "v_cvt_pk_bf16_f32 v132, v91, v95\n\t"
        "buffer_load_dwordx4 v[88:91], v171, %[rw], 0 offen\n\t"
        "buffer_load_dwordx4 v[92:95], v171, %[rw], s48 offen\n\t"
        "s_waitcnt vmcnt(14)\n\t"
        "v_cvt_pk_bf16_f32 v121, v96, v100\n\t"
        "v_cvt_pk_bf16_f32 v125, v97, v101\n\t"
        "v_cvt_pk_bf16_f32 v129, v98, v102\n\t"
        "v_cvt_pk_bf16_f32 v133, v99, v103\n\t"
        "buffer_load_dwordx4 v[96:99], v171, %[rw], s49 offen\n\t"
        "buffer_load_dwordx4 v[100:103], v171, %[rw], s50 offen\n\t"
        "s_waitcnt vmcnt(14)\n\t"
        "v_cvt_pk_bf16_f32 v122, v104, v108\n\t"
        "v_cvt_pk_bf16_f32 v126, v105, v109\n\t"
        "v_cvt_pk_bf16_f32 v130, v106, v110\n\t"
        "v_cvt_pk_bf16_f32 v134, v107, v111\n\t"
        "buffer_load_dwordx4 v[104:107], v171, %[rw], s51 offen\n\t"
        "buffer_load_dwordx4 v[108:111], v171, %[rw], s52 offen\n\t"
        "s_waitcnt vmcnt(14)\n\t"
        "v_cvt_pk_bf16_f32 v123, v112, v116\n\t"
        "v_cvt_pk_bf16_f32 v127, v113, v117\n\t"
        "v_cvt_pk_bf16_f32 v131, v114, v118\n\t"
        "v_cvt_pk_bf16_f32 v135, v115, v119\n\t"
        "ds_write_b128 v173, v[120:123] offset:0\n\t"
        "ds_write_b128 v173, v[124:127] offset:1024\n\t"
        "ds_write_b128 v173, v[128:131] offset:2048\n\t"
        "ds_write_b128 v173, v[132:135] offset:3072\n\t"
        "buffer_load_dwordx4 v[112:115], v171, %[rw], s53 offen\n\t"
        "buffer_load_dwordx4 v[116:119], v171, %[rw], s54 offen\n\t"
        "s_waitcnt lgkmcnt(0)\n\t"
        "s_barrier\n\t"
        "s_mov_b32 s59, 0\n\t"
        "s_mov_b32 s56, 4\n\t"
        "v_readlane_b32 s57, %[otx], 4\n\t"
        "v_readlane_b32 s58, %[otw], 4\n\t"
        "v_add_u32 v173, s59, %[wl]\n\t"
        "s_add_u32 s59, s59, 49152\n\t"
        "s_cmp_eq_u32 s59, 147456\n\t"
        "s_cselect_b32 s59, 0, s59\n\t"
        "s_add_u32 s60, s59, %[db]\n\t"
        "s_add_u32 s68, s57, 4096\n\t"
        "s_mov_b32 m0, s60\n\t"
        "s_nop 0\n\t"
        "LLOOP%=:\n\t"
        "buffer_load_dwordx4 %[dv], %[rs], s57 offen offset:0 sc0 sc1 lds\n\t"
        "s_mov_b32 s40, s58\n\t"
        "buffer_load_dwordx4 %[dv], %[rs], s57 offen offset:1024 sc0 sc1 lds\n\t"
        "s_add_u32 s41, s58, s48\n\t"
        "s_waitcnt vmcnt(8)\n\t"
        "v_cvt_pk_bf16_f32 v120, v88, v92\n\t"
        "v_cvt_pk_bf16_f32 v124, v89, v93\n\t"
        "v_cvt_pk_bf16_f32 v128, v90, v94\n\t"
        "v_cvt_pk_bf16_f32 v132, v91, v95\n\t"
        "buffer_load_dwordx4 %[dv], %[rs], s57 offen offset:2048 sc0 sc1 lds\n\t"
        "s_add_u32 s42, s58, s49\n\t"
        "buffer_load_dwordx4 %[dv], %[rs], s57 offen offset:3072 sc0 sc1 lds\n\t"
        "s_add_u32 s43, s58, s50\n\t"
        "s_waitcnt vmcnt(8)\n\t"
        "v_cvt_pk_bf16_f32 v121, v96, v100\n\t"
        "v_cvt_pk_bf16_f32 v125, v97, v101\n\t"
        "v_cvt_pk_bf16_f32 v129, v98, v102\n\t"
        "v_cvt_pk_bf16_f32 v133, v99, v103\n\t"
        "s_add_u32 m0, s60, 4096\n\t"
        "s_nop 0\n\t"
        "buffer_load_dwordx4 %[dv], %[rs], s68 offen offset:0 sc0 sc1 lds\n\t"
        "s_add_u32 s44, s58, s51\n\t"
        "buffer_load_dwordx4 %[dv], %[rs], s68 offen offset:1024 sc0 sc1 lds\n\t"
        "s_add_u32 s45, s58, s52\n\t"
        "s_waitcnt vmcnt(8)\n\t"
        "v_cvt_pk_bf16_f32 v122, v104, v108\n\t"
        "v_cvt_pk_bf16_f32 v126, v105, v109\n\t"
        "v_cvt_pk_bf16_f32 v130, v106, v110\n\t"
        "v_cvt_pk_bf16_f32 v134, v107, v111\n\t"
        "buffer_load_dwordx4 %[dv], %[rs], s68 offen offset:2048 sc0 sc1 lds\n\t"
        "s_add_u32 s46, s58, s53\n\t"
        "buffer_load_dwordx4 %[dv], %[rs], s68 offen offset:3072 sc0 sc1 lds\n\t"
        "s_add_u32 s47, s58, s54\n\t"
        "s_waitcnt vmcnt(8)\n\t"
        "v_cvt_pk_bf16_f32 v123, v112, v116\n\t"
        "v_cvt_pk_bf16_f32 v127, v113, v117\n\t"
        "v_cvt_pk_bf16_f32 v131, v114, v118\n\t"
        "v_cvt_pk_bf16_f32 v135, v115, v119\n\t"
        "s_add_u32 s56, s56, 1\n\t"
        "buffer_load_dwordx4 v[88:91], %[wv], %[rw], s40 offen\n\t"
        "buffer_load_dwordx4 v[92:95], %[wv], %[rw], s41 offen\n\t"
        "buffer_load_dwordx4 v[96:99], %[wv], %[rw], s42 offen\n\t"
        "buffer_load_dwordx4 v[100:103], %[wv], %[rw], s43 offen\n\t"
        "buffer_load_dwordx4 v[104:107], %[wv], %[rw], s44 offen\n\t"
        "v_readlane_b32 s57, %[otx], s56\n\t"
        "v_readlane_b32 s58, %[otw], s56\n\t"
        "s_mov_b32 s67, s59\n\t"
        "s_add_u32 s59, s59, 49152\n\t"
        "s_cmp_eq_u32 s59, 147456\n\t"
        "s_cselect_b32 s59, 0, s59\n\t"
        "s_add_u32 s60, s59, %[db]\n\t"
        "s_add_u32 s68, s57, 4096\n\t"
        "s_mov_b32 m0, s60\n\t"
        "buffer_load_dwordx4 v[108:111], %[wv], %[rw], s45 offen\n\t"
        "buffer_load_dwordx4 v[112:115], %[wv], %[rw], s46 offen\n\t"
        "ds_write_b128 v173, v[120:123] offset:0\n\t"
        "ds_write_b128 v173, v[124:127] offset:1024\n\t"
        "ds_write_b128 v173, v[128:131] offset:2048\n\t"
        "ds_write_b128 v173, v[132:135] offset:3072\n\t"
        "buffer_load_dwordx4 v[116:119], %[wv], %[rw], s47 offen\n\t"
        "v_add_u32 v173, s67, %[wl]\n\t"
        "s_cmp_lt_u32 s56, 16\n\t"
        "s_waitcnt lgkmcnt(0)\n\t"
        "s_barrier\n\t"
        "s_cbranch_scc1 LLOOP%=\n\t"
        "v_mov_b32 v24, -1\n\t"
        "v_mov_b32 v25, -1\n\t"
        "v_mov_b32 v26, -1\n\t"
        "v_mov_b32 v27, -1\n\t"
        "v_mov_b32 v28, 1\n\t"
        "buffer_store_dwordx4 v[24:27], %[pv], %[rs], 0 offen offset:0 sc0 sc1\n\t"
        "buffer_store_dwordx4 v[24:27], %[pv], %[rs], 0 offen offset:1024 sc0 sc1\n\t"
        "buffer_store_dwordx4 v[24:27], %[pv2], %[rs], 0 offen sc0 sc1\n\t"
        "s_waitcnt vmcnt(9)\n\t"
        "v_cvt_pk_bf16_f32 v120, v88, v92\n\t"
        "v_cvt_pk_bf16_f32 v124, v89, v93\n\t"
        "v_cvt_pk_bf16_f32 v128, v90, v94\n\t"
        "v_cvt_pk_bf16_f32 v132, v91, v95\n\t"
        "s_waitcnt vmcnt(7)\n\t"
        "v_cvt_pk_bf16_f32 v121, v96, v100\n\t"
        "v_cvt_pk_bf16_f32 v125, v97, v101\n\t"
        "v_cvt_pk_bf16_f32 v129, v98, v102\n\t"
        "v_cvt_pk_bf16_f32 v133, v99, v103\n\t"
        "s_waitcnt vmcnt(5)\n\t"
        "v_cvt_pk_bf16_f32 v122, v104, v108\n\t"
        "v_cvt_pk_bf16_f32 v126, v105, v109\n\t"
        "v_cvt_pk_bf16_f32 v130, v106, v110\n\t"
        "v_cvt_pk_bf16_f32 v134, v107, v111\n\t"
        "s_waitcnt vmcnt(3)\n\t"
        "v_cvt_pk_bf16_f32 v123, v112, v116\n\t"
        "v_cvt_pk_bf16_f32 v127, v113, v117\n\t"
        "v_cvt_pk_bf16_f32 v131, v114, v118\n\t"
        "v_cvt_pk_bf16_f32 v135, v115, v119\n\t"
        "ds_write_b128 v173, v[120:123] offset:0\n\t"
        "ds_write_b128 v173, v[124:127] offset:1024\n\t"
        "ds_write_b128 v173, v[128:131] offset:2048\n\t"
        "ds_write_b128 v173, v[132:135] offset:3072\n\t"
        "s_waitcnt lgkmcnt(0)\n\t"
        "s_barrier\n\t"
        "s_waitcnt vmcnt(0)\n\t"
        "ds_write_b32 %[mf], v28\n\t"
        "s_waitcnt lgkmcnt(0)\n\t"
        "s_mov_b32 m0, s66\n\t"
        "s_setprio 0\n\t"
        : [call] "=&s"(call), "+{v[136:167]}"(cvr)
        : [xv] "v"(xv), [wv] "v"(wv), [xw] "v"(xw), [wl] "v"(wl), [otx] "v"(otx), [otw] "v"(otw), [cs] "v"(cs), [dv] "v"(dv),
          [fo] "v"(fo), [po] "v"(po), [pv] "v"(pv), [pv2] "v"(pv2), [mf] "v"(mf), [rx] "s"(rx), [rw] "s"(rw), [rs] "s"(rws), [g0] "s"(g0), [cb] "s"(cb), [db] "s"(db),
          [dl] "s"(dl), [ct] "s"(ct), [tm] "s"(tm)
        : "memory", "scc", "vcc", "v24", "v25", "v26", "v27", "v28", "v29", "v30", "v31", "v32", "v33", "v34", "v35", "v36", "v37", "v38", "v39", "v40", "v41", "v42", "v43", "v44", "v45", "v46", "v47", "v48", "v49", "v50", "v51", "v52", "v53", "v54", "v55", "v56", "v57", "v58", "v59", "v60", "v61", "v62", "v63", "v64", "v65", "v66", "v67", "v68", "v69", "v70", "v71", "v72", "v73", "v74", "v75", "v76", "v77", "v78", "v79", "v80", "v81", "v82", "v83", "v84", "v85", "v86", "v87", "v88", "v89", "v90", "v91", "v92", "v93", "v94", "v95", "v96", "v97", "v98", "v99", "v100", "v101", "v102", "v103", "v104", "v105", "v106", "v107", "v108", "v109", "v110", "v111", "v112", "v113", "v114", "v115", "v116", "v117", "v118", "v119", "v120", "v121", "v122", "v123", "v124", "v125", "v126", "v127", "v128", "v129", "v130", "v131", "v132", "v133", "v134", "v135", "v168", "v169", "v170", "v171", "v172", "v173", "v174", "v175", "s40", "s41", "s42", "s43", "s44", "s45", "s46", "s47", "s48", "s49", "s50", "s51", "s52", "s53", "s54", "s55", "s56", "s57", "s58", "s59", "s60", "s61", "s62", "s63", "s64", "s65", "s66", "s67", "s68");

        }
        const int lane = lane_id();
        const unsigned t0 = (unsigned)rt();
        float y[32];
        bool bad = false;
        const bool paced = (ni >> 2) != rank;
        u64 slot;
        const unsigned xoff = WS_XFLAG + (b * 4 + wave) * 64 + (lane & 7) * 4;
        const unsigned g = 0;
        f32x32 yv;
        unsigned gbad, gk;
        {
            const unsigned l16 = lane * 16, of = ldsb + L_CTL + (8 + wave) * 4, oa = ldsb + L_OWN + wave * 8192 + l16;
            const unsigned pbase = WS_PART + b * (SPLIT - 1) * 16384 + wave * 4096, nobook = __builtin_amdgcn_readfirstlane((paced && wave == 0) ? 0u : 1u), pc = WS_PACE + (ni >> 2) * 128;
        asm volatile(
        "v_mov_b32 v128, 0\n\t"
        "v_mov_b32 v129, 0\n\t"
        "v_mov_b32 v130, 0\n\t"
        "v_mov_b32 v131, 0\n\t"
        "v_mov_b32 v132, 0\n\t"
        "v_mov_b32 v133, 0\n\t"
        "v_mov_b32 v134, 0\n\t"
        "v_mov_b32 v135, 0\n\t"
        "v_mov_b32 v136, 0\n\t"
        "v_mov_b32 v137, 0\n\t"
        "v_mov_b32 v138, 0\n\t"
        "v_mov_b32 v139, 0\n\t"
        "v_mov_b32 v140, 0\n\t"
        "v_mov_b32 v141, 0\n\t"
        "v_mov_b32 v142, 0\n\t"
        "v_mov_b32 v143, 0\n\t"
        "v_mov_b32 v144, 0\n\t"
        "v_mov_b32 v145, 0\n\t"
        "v_mov_b32 v146, 0\n\t"
        "v_mov_b32 v147, 0\n\t"
        "v_mov_b32 v148, 0\n\t"
        "v_mov_b32 v149, 0\n\t"
        "v_mov_b32 v150, 0\n\t"
        "v_mov_b32 v151, 0\n\t"
        "v_mov_b32 v152, 0\n\t"
        "v_mov_b32 v153, 0\n\t"
        "v_mov_b32 v154, 0\n\t"
        "v_mov_b32 v155, 0\n\t"
        "v_mov_b32 v156, 0\n\t"
        "v_mov_b32 v157, 0\n\t"
        "v_mov_b32 v158, 0\n\t"
        "v_mov_b32 v159, 0\n\t"
        "v_mov_b32 v104, %[g]\n\t"
        "s_mov_b32 s40, 31\n\t"
        "s_add_u32 s48, %[pb], 16384\n\t"
        "s_add_u32 s49, %[pb], 32768\n\t"
        "s_mov_b32 %[bad], 0\n\t"
        "GL%=:\n\t"
        "s_waitcnt vmcnt(0)\n\t"
        "v_cmp_le_u32 vcc, %[call], v104\n\t"
        "ds_read_b32 v106, %[of]\n\t"
        "s_nop 1\n\t"
        "s_and_b32 s41, vcc_lo, s40\n\t"
        "s_and_b32 s41, s41, 23\n\t"
        "s_andn2_b32 s40, s40, s41\n\t"
        "s_bitcmp0_b32 s41, 0\n\t"
        "s_cbranch_scc1 GA0%=\n\t"
        "buffer_load_dwordx4 v[24:27], %[l16], %[rs], %[pb] offen offset:0 sc1\n\t"
        "buffer_load_dwordx4 v[28:31], %[l16], %[rs], %[pb] offen offset:1024 sc1\n\t"
        "buffer_load_dwordx4 v[32:35], %[l16], %[rs], %[pb] offen offset:2048 sc1\n\t"
        "buffer_load_dwordx4 v[36:39], %[l16], %[rs], %[pb] offen offset:3072 sc1\n\t"
        "GA0%=:\n\t"
        "s_bitcmp0_b32 s41, 1\n\t"
        "s_cbranch_scc1 GA1%=\n\t"
        "buffer_load_dwordx4 v[40:43], %[l16], %[rs], s48 offen offset:0 sc1\n\t"
        "buffer_load_dwordx4 v[44:47], %[l16], %[rs], s48 offen offset:1024 sc1\n\t"
        "buffer_load_dwordx4 v[48:51], %[l16], %[rs], s48 offen offset:2048 sc1\n\t"
        "buffer_load_dwordx4 v[52:55], %[l16], %[rs], s48 offen offset:3072 sc1\n\t"
        "GA1%=:\n\t"
        "s_bitcmp0_b32 s41, 2\n\t"
        "s_cbranch_scc1 GA2%=\n\t"
        "buffer_load_dwordx4 v[56:59], %[l16], %[rs], s49 offen offset:0 sc1\n\t"
        "buffer_load_dwordx4 v[60:63], %[l16], %[rs], s49 offen offset:1024 sc1\n\t"
        "buffer_load_dwordx4 v[64:67], %[l16], %[rs], s49 offen offset:2048 sc1\n\t"
        "buffer_load_dwordx4 v[68:71], %[l16], %[rs], s49 offen offset:3072 sc1\n\t"
        "GA2%=:\n\t"
        "buffer_load_dword v104, %[xo], %[rs], 0 offen sc0 sc1\n\t"
        "s_waitcnt lgkmcnt(0)\n\t"
        "v_readfirstlane_b32 s44, v106\n\t"
        "s_nop 3\n\t"
        "s_lshl_b32 s44, s44, 3\n\t"
        "s_and_b32 s44, s44, s40\n\t"
        "s_cbranch_scc0 GB%=\n\t"
        "ds_read_b128 v[72:75], %[oa] offset:0\n\t"
        "ds_read_b128 v[76:79], %[oa] offset:1024\n\t"
        "ds_read_b128 v[80:83], %[oa] offset:2048\n\t"
        "ds_read_b128 v[84:87], %[oa] offset:3072\n\t"
        "ds_read_b128 v[88:91], %[oa] offset:4096\n\t"
        "ds_read_b128 v[92:95], %[oa] offset:5120\n\t"
        "ds_read_b128 v[96:99], %[oa] offset:6144\n\t"
        "ds_read_b128 v[100:103], %[oa] offset:7168\n\t"
        "s_andn2_b32 s40, s40, 8\n\t"
        "s_waitcnt lgkmcnt(7)\n\t"
        "v_pk_add_f32 v[128:129], v[128:129], v[72:73]\n\t"
        "v_pk_add_f32 v[130:131], v[130:131], v[74:75]\n\t"
        "s_waitcnt lgkmcnt(6)\n\t"
        "v_pk_add_f32 v[132:133], v[132:133], v[76:77]\n\t"
        "v_pk_add_f32 v[134:135], v[134:135], v[78:79]\n\t"
        "s_waitcnt lgkmcnt(5)\n\t"
        "v_pk_add_f32 v[136:137], v[136:137], v[80:81]\n\t"
        "v_pk_add_f32 v[138:139], v[138:139], v[82:83]\n\t"
        "s_waitcnt lgkmcnt(4)\n\t"
        "v_pk_add_f32 v[140:141], v[140:141], v[84:85]\n\t"
        "v_pk_add_f32 v[142:143], v[142:143], v[86:87]\n\t"
        "s_waitcnt lgkmcnt(3)\n\t"
        "v_pk_add_f32 v[144:145], v[144:145], v[88:89]\n\t"
        "v_pk_add_f32 v[146:147], v[146:147], v[90:91]\n\t"
        "s_waitcnt lgkmcnt(2)\n\t"
        "v_pk_add_f32 v[148:149], v[148:149], v[92:93]\n\t"
        "v_pk_add_f32 v[150:151], v[150:151], v[94:95]\n\t"
        "s_waitcnt lgkmcnt(1)\n\t"
        "v_pk_add_f32 v[152:153], v[152:153], v[96:97]\n\t"
        "v_pk_add_f32 v[154:155], v[154:155], v[98:99]\n\t"
        "s_waitcnt lgkmcnt(0)\n\t"
        "v_pk_add_f32 v[156:157], v[156:157], v[100:101]\n\t"
        "v_pk_add_f32 v[158:159], v[158:159], v[102:103]\n\t"
        "GB%=:\n\t"
        "s_or_b32 s44, s40, %[nb]\n\t"
        "s_cbranch_scc1 GC%=\n\t"
        "s_memrealtime s[44:45]\n\t"
        "s_mov_b32 s50, 0\n\t"
        "v_cmp_lt_f32 vcc, 0, v128\n\t"
        "s_nop 3\n\t"
        "s_bcnt1_i32_b64 s46, vcc\n\t"
        "s_add_u32 s50, s50, s46\n\t"
        "v_cmp_lt_f32 vcc, 0, v137\n\t"
        "s_nop 3\n\t"
        "s_bcnt1_i32_b64 s46, vcc\n\t"
        "s_add_u32 s50, s50, s46\n\t"
        "v_cmp_lt_f32 vcc, 0, v146\n\t"
        "s_nop 3\n\t"
        "s_bcnt1_i32_b64 s46, vcc\n\t"
        "s_add_u32 s50, s50, s46\n\t"
        "v_cmp_lt_f32 vcc, 0, v155\n\t"
        "s_nop 3\n\t"
        "s_bcnt1_i32_b64 s46, vcc\n\t"
        "s_add_u32 s50, s50, s46\n\t"
        "s_mul_i32 s50, s50, 62\n\t"
        "s_add_u32 s50, s50, 1536\n\t"
        "s_mul_i32 s50, s50, %[pace]\n\t"
        "s_lshr_b32 s50, s50, 10\n\t"
        "s_waitcnt lgkmcnt(0)\n\t"
        "s_add_u32 s44, s44, %[rdt]\n\t"
        "s_addc_u32 s45, s45, 0\n\t"
        "s_lshl_b64 s[44:45], s[44:45], 4\n\t"
        "v_mov_b32 v112, s44\n\t"
        "v_mov_b32 v113, s45\n\t"
        "v_mov_b32 v114, s50\n\t"
        "v_mov_b32 v115, 0\n\t"
        "s_mov_b64 exec, 1\n\t"
        "buffer_atomic_umax_x2 v[112:113], off, %[rs], %[pc]\n\t"
        "buffer_atomic_add_x2 v[114:115], off, %[rs], %[pc] sc0\n\t"
        "s_mov_b64 exec, -1\n\t"
        "s_waitcnt vmcnt(3)\n\t"
        "s_branch GD%=\n\t"
        "GC%=:\n\t"
        "s_waitcnt vmcnt(1)\n\t"
        "GD%=:\n\t"
        "s_bitcmp0_b32 s41, 0\n\t"
        "s_cbranch_scc1 GE0%=\n\t"
        "v_lshlrev_b32 v106, 16, v24\n\t"
        "v_and_b32 v107, 0xffff0000, v24\n\t"
        "v_pk_add_f32 v[128:129], v[128:129], v[106:107]\n\t"
        "v_lshlrev_b32 v108, 16, v25\n\t"
        "v_and_b32 v109, 0xffff0000, v25\n\t"
        "v_pk_add_f32 v[130:131], v[130:131], v[108:109]\n\t"
        "v_lshlrev_b32 v106, 16, v26\n\t"
        "v_and_b32 v107, 0xffff0000, v26\n\t"
        "v_pk_add_f32 v[132:133], v[132:133], v[106:107]\n\t"
        "v_lshlrev_b32 v108, 16, v27\n\t"
        "v_and_b32 v109, 0xffff0000, v27\n\t"
        "v_pk_add_f32 v[134:135], v[134:135], v[108:109]\n\t"
        "v_lshlrev_b32 v106, 16, v28\n\t"
        "v_and_b32 v107, 0xffff0000, v28\n\t"
        "v_pk_add_f32 v[136:137], v[136:137], v[106:107]\n\t"
        "v_lshlrev_b32 v108, 16, v29\n\t"
        "v_and_b32 v109, 0xffff0000, v29\n\t"
        "v_pk_add_f32 v[138:139], v[138:139], v[108:109]\n\t"
        "v_lshlrev_b32 v106, 16, v30\n\t"
        "v_and_b32 v107, 0xffff0000, v30\n\t"
        "v_pk_add_f32 v[140:141], v[140:141], v[106:107]\n\t"
        "v_lshlrev_b32 v108, 16, v31\n\t"
        "v_and_b32 v109, 0xffff0000, v31\n\t"
        "v_pk_add_f32 v[142:143], v[142:143], v[108:109]\n\t"
        "v_lshlrev_b32 v106, 16, v32\n\t"
        "v_and_b32 v107, 0xffff0000, v32\n\t"
        "v_pk_add_f32 v[144:145], v[144:145], v[106:107]\n\t"
        "v_lshlrev_b32 v108, 16, v33\n\t"
        "v_and_b32 v109, 0xffff0000, v33\n\t"
        "v_pk_add_f32 v[146:147], v[146:147], v[108:109]\n\t"
        "v_lshlrev_b32 v106, 16, v34\n\t"
        "v_and_b32 v107, 0xffff0000, v34\n\t"
        "v_pk_add_f32 v[148:149], v[148:149], v[106:107]\n\t"
        "v_lshlrev_b32 v108, 16, v35\n\t"
        "v_and_b32 v109, 0xffff0000, v35\n\t"
        "v_pk_add_f32 v[150:151], v[150:151], v[108:109]\n\t"
        "v_lshlrev_b32 v106, 16, v36\n\t"
        "v_and_b32 v107, 0xffff0000, v36\n\t"
        "v_pk_add_f32 v[152:153], v[152:153], v[106:107]\n\t"
        "v_lshlrev_b32 v108, 16, v37\n\t"
        "v_and_b32 v109, 0xffff0000, v37\n\t"
        "v_pk_add_f32 v[154:155], v[154:155], v[108:109]\n\t"
        "v_lshlrev_b32 v106, 16, v38\n\t"
        "v_and_b32 v107, 0xffff0000, v38\n\t"
        "v_pk_add_f32 v[156:157], v[156:157], v[106:107]\n\t"
        "v_lshlrev_b32 v108, 16, v39\n\t"
        "v_and_b32 v109, 0xffff0000, v39\n\t"
        "v_pk_add_f32 v[158:159], v[158:159], v[108:109]\n\t"
        "GE0%=:\n\t"
        "s_bitcmp0_b32 s41, 1\n\t"
        "s_cbranch_scc1 GE1%=\n\t"
        "v_lshlrev_b32 v106, 16, v40\n\t"
        "v_and_b32 v107, 0xffff0000, v40\n\t"
        "v_pk_add_f32 v[128:129], v[128:129], v[106:107]\n\t"
        "v_lshlrev_b32 v108, 16, v41\n\t"
        "v_and_b32 v109, 0xffff0000, v41\n\t"
        "v_pk_add_f32 v[130:131], v[130:131], v[108:109]\n\t"
        "v_lshlrev_b32 v106, 16, v42\n\t"
        "v_and_b32 v107, 0xffff0000, v42\n\t"
        "v_pk_add_f32 v[132:133], v[132:133], v[106:107]\n\t"
        "v_lshlrev_b32 v108, 16, v43\n\t"
        "v_and_b32 v109, 0xffff0000, v43\n\t"
        "v_pk_add_f32 v[134:135], v[134:135], v[108:109]\n\t"
        "v_lshlrev_b32 v106, 16, v44\n\t"
        "v_and_b32 v107, 0xffff0000, v44\n\t"
        "v_pk_add_f32 v[136:137], v[136:137], v[106:107]\n\t"
        "v_lshlrev_b32 v108, 16, v45\n\t"
        "v_and_b32 v109, 0xffff0000, v45\n\t"
        "v_pk_add_f32 v[138:139], v[138:139], v[108:109]\n\t"
        "v_lshlrev_b32 v106, 16, v46\n\t"
        "v_and_b32 v107, 0xffff0000, v46\n\t"
        "v_pk_add_f32 v[140:141], v[140:141], v[106:107]\n\t"
        "v_lshlrev_b32 v108, 16, v47\n\t"
        "v_and_b32 v109, 0xffff0000, v47\n\t"
        "v_pk_add_f32 v[142:143], v[142:143], v[108:109]\n\t"
        "v_lshlrev_b32 v106, 16, v48\n\t"
        "v_and_b32 v107, 0xffff0000, v48\n\t"
        "v_pk_add_f32 v[144:145], v[144:145], v[106:107]\n\t"
        "v_lshlrev_b32 v108, 16, v49\n\t"
        "v_and_b32 v109, 0xffff0000, v49\n\t"
        "v_pk_add_f32 v[146:147], v[146:147], v[108:109]\n\t"
        "v_lshlrev_b32 v106, 16, v50\n\t"
        "v_and_b32 v107, 0xffff0000, v50\n\t"
        "v_pk_add_f32 v[148:149], v[148:149], v[106:107]\n\t"
        "v_lshlrev_b32 v108, 16, v51\n\t"
        "v_and_b32 v109, 0xffff0000, v51\n\t"
        "v_pk_add_f32 v[150:151], v[150:151], v[108:109]\n\t"
        "v_lshlrev_b32 v106, 16, v52\n\t"
        "v_and_b32 v107, 0xffff0000, v52\n\t"
        "v_pk_add_f32 v[152:153], v[152:153], v[106:107]\n\t"
        "v_lshlrev_b32 v108, 16, v53\n\t"
        "v_and_b32 v109, 0xffff0000, v53\n\t"
        "v_pk_add_f32 v[154:155], v[154:155], v[108:109]\n\t"
        "v_lshlrev_b32 v106, 16, v54\n\t"
        "v_and_b32 v107, 0xffff0000, v54\n\t"
        "v_pk_add_f32 v[156:157], v[156:157], v[106:107]\n\t"
        "v_lshlrev_b32 v108, 16, v55\n\t"
        "v_and_b32 v109, 0xffff0000, v55\n\t"
        "v_pk_add_f32 v[158:159], v[158:159], v[108:109]\n\t"
        "GE1%=:\n\t"
        "s_bitcmp0_b32 s41, 2\n\t"
        "s_cbranch_scc1 GE2%=\n\t"
        "v_lshlrev_b32 v106, 16, v56\n\t"
        "v_and_b32 v107, 0xffff0000, v56\n\t"
        "v_pk_add_f32 v[128:129], v[128:129], v[106:107]\n\t"
        "v_lshlrev_b32 v108, 16, v57\n\t"
        "v_and_b32 v109, 0xffff0000, v57\n\t"
        "v_pk_add_f32 v[130:131], v[130:131], v[108:109]\n\t"
        "v_lshlrev_b32 v106, 16, v58\n\t"
        "v_and_b32 v107, 0xffff0000, v58\n\t"
        "v_pk_add_f32 v[132:133], v[132:133], v[106:107]\n\t"
        "v_lshlrev_b32 v108, 16, v59\n\t"
        "v_and_b32 v109, 0xffff0000, v59\n\t"
        "v_pk_add_f32 v[134:135], v[134:135], v[108:109]\n\t"
        "v_lshlrev_b32 v106, 16, v60\n\t"
        "v_and_b32 v107, 0xffff0000, v60\n\t"
        "v_pk_add_f32 v[136:137], v[136:137], v[106:107]\n\t"
        "v_lshlrev_b32 v108, 16, v61\n\t"
        "v_and_b32 v109, 0xffff0000, v61\n\t"
        "v_pk_add_f32 v[138:139], v[138:139], v[108:109]\n\t"
        "v_lshlrev_b32 v106, 16, v62\n\t"
        "v_and_b32 v107, 0xffff0000, v62\n\t"
        "v_pk_add_f32 v[140:141], v[140:141], v[106:107]\n\t"
        "v_lshlrev_b32 v108, 16, v63\n\t"
        "v_and_b32 v109, 0xffff0000, v63\n\t"
        "v_pk_add_f32 v[142:143], v[142:143], v[108:109]\n\t"
        "v_lshlrev_b32 v106, 16, v64\n\t"
        "v_and_b32 v107, 0xffff0000, v64\n\t"
        "v_pk_add_f32 v[144:145], v[144:145], v[106:107]\n\t"
        "v_lshlrev_b32 v108, 16, v65\n\t"
        "v_and_b32 v109, 0xffff0000, v65\n\t"
        "v_pk_add_f32 v[146:147], v[146:147], v[108:109]\n\t"
        "v_lshlrev_b32 v106, 16, v66\n\t"
        "v_and_b32 v107, 0xffff0000, v66\n\t"
        "v_pk_add_f32 v[148:149], v[148:149], v[106:107]\n\t"
        "v_lshlrev_b32 v108, 16, v67\n\t"
        "v_and_b32 v109, 0xffff0000, v67\n\t"
        "v_pk_add_f32 v[150:151], v[150:151], v[108:109]\n\t"
        "v_lshlrev_b32 v106, 16, v68\n\t"
        "v_and_b32 v107, 0xffff0000, v68\n\t"
        "v_pk_add_f32 v[152:153], v[152:153], v[106:107]\n\t"
        "v_lshlrev_b32 v108, 16, v69\n\t"
        "v_and_b32 v109, 0xffff0000, v69\n\t"
        "v_pk_add_f32 v[154:155], v[154:155], v[108:109]\n\t"
        "v_lshlrev_b32 v106, 16, v70\n\t"
        "v_and_b32 v107, 0xffff0000, v70\n\t"
        "v_pk_add_f32 v[156:157], v[156:157], v[106:107]\n\t"
        "v_lshlrev_b32 v108, 16, v71\n\t"
        "v_and_b32 v109, 0xffff0000, v71\n\t"
        "v_pk_add_f32 v[158:159], v[158:159], v[108:109]\n\t"
        "GE2%=:\n\t"
        "s_cmp_eq_u32 s40, 0\n\t"
        "s_cbranch_scc1 GX%=\n\t"
        "s_memrealtime s[44:45]\n\t"
        "s_waitcnt lgkmcnt(0)\n\t"
        "s_sub_u32 s44, s44, %[t0]\n\t"
        "s_cmp_lt_u32 s44, 5000\n\t"
        "s_cbranch_scc1 GL%=\n\t"
        "buffer_load_dword v106, off, %[rs], %[tm] sc1\n\t"
        "s_sleep 8\n\t"
        "s_waitcnt vmcnt(0)\n\t"
        "v_readfirstlane_b32 s42, v106\n\t"
        "s_nop 3\n\t"
        "s_cmp_eq_u32 s42, %[call]\n\t"
        "s_cbranch_scc1 GF%=\n\t"
        "s_cmp_lt_u32 s44, %[dl]\n\t"
        "s_cbranch_scc1 GL%=\n\t"
        "GF%=:\n\t"
        "s_mov_b32 %[bad], 1\n\t"
        "GX%=:\n\t"
        : "=&{v[128:159]}"(yv), "=&{v[114:115]}"(slot), "=&{v104}"(gk), [bad] "=&s"(gbad)
        : [g] "v"(g), [xo] "v"(xoff), [l16] "v"(l16), [of] "v"(of), [oa] "v"(oa), [rs] "s"(rws), [pb] "s"(pbase), [call] "s"(call), [nb] "s"(nobook),
          [pace] "s"(PACE), [rdt] "s"(READY_T), [pc] "s"(pc), [t0] "s"(t0), [dl] "s"(DEADLINE_TICKS), [tm] "s"(WS_TMO)
        : "memory", "scc", "vcc", "v24", "v25", "v26", "v27", "v28", "v29", "v30", "v31", "v32", "v33", "v34", "v35", "v36", "v37", "v38", "v39", "v40", "v41", "v42", "v43", "v44", "v45", "v46", "v47", "v48", "v49", "v50", "v51", "v52", "v53", "v54", "v55", "v56", "v57", "v58", "v59", "v60", "v61", "v62", "v63", "v64", "v65", "v66", "v67", "v68", "v69", "v70", "v71", "v72", "v73", "v74", "v75", "v76", "v77", "v78", "v79", "v80", "v81", "v82", "v83", "v84", "v85", "v86", "v87", "v88", "v89", "v90", "v91", "v92", "v93", "v94", "v95", "v96", "v97", "v98", "v99", "v100", "v101", "v102", "v103", "v105", "v106", "v107", "v108", "v109", "v110", "v111", "v112", "v113", "s40", "s41", "s42", "s43", "s44", "s45", "s46", "s47", "s48", "s49", "s50", "s51");

        }
        if (gbad) { bad = true; if (lane == 0) __hip_atomic_store(tmo, call, __ATOMIC_RELAXED, __HIP_MEMORY_SCOPE_AGENT); }
#pragma unroll
        for (int e = 0; e < 32; ++e) y[e] = yv[e];
        unsigned mask = 0, pk[16], n2[16];
#pragma unroll
        for (int k = 0; k < 16; ++k) {
            const unsigned c = pk_bf16(y[2 * k], y[2 * k + 1]);
            asm("v_pk_max_i16 %0, %3, 0\n\tv_pk_min_u16 %1, %0, %4\n\tv_lshl_or_b32 %2, %1, %5, %2\n\tv_add_u32 %1, %1, %1"
                : "=&v"(pk[k]), "=&v"(n2[k]), "+v"(mask) : "v"(c), "s"(0x00010001u), "i"(k));
        }
        mask &= 0x7fffffffu;
        asm volatile("s_waitcnt vmcnt(0)" : "+{v[114:115]}"(slot), "+{v104}"(gk), "+v"(mask) :: "memory");
        if (wave == 0 && lane == 0) ctl[5] = (bad ? t0 : (unsigned)(slot >> 4)) | 1u;
        unsigned total;
        char* const pb = lds + L_PK + (4 + wave) * PK;
        unsigned pos = ldsb + L_PK + (4 + wave) * PK + wave_scan(__builtin_popcount(mask) + 1, total) * 2u;
        total = 256u + total * 2u;
        if (lane == 0) *(u32x4*)(pb + (total & ~15u)) = u32x4{0, 0, 0, 0};
        *(unsigned*)(pb + lane * 4) = mask;
#pragma unroll
        for (int k = 0; k < 16; ++k)
            asm volatile("ds_write_b16 %0, %1 offset:256\n\tv_add_u32_sdwa %0, %0, %2 dst_sel:DWORD dst_unused:UNUSED_PAD src0_sel:DWORD src1_sel:WORD_0\n\t"
                         "ds_write_b16_d16_hi %0, %1 offset:256\n\tv_add_u32_sdwa %0, %0, %2 dst_sel:DWORD dst_unused:UNUSED_PAD src0_sel:DWORD src1_sel:WORD_1"
                         : "+v"(pos) : "v"(pk[k]), "v"(n2[k]) : "memory");
        asm volatile("" ::: "memory");
        u32x4 d[3];
#pragma unroll
        for (int u = 0; u < 3; ++u) d[u] = *(const u32x4*)(pb + u * 1024 + lane * 16);
        unsigned go = paced ? __builtin_amdgcn_readfirstlane(ctl[5]) : 0u;
        auto send = [&]() {
            const __amdgpu_buffer_rsrc_t rd = mk_rsrc(dst_ws, bad ? 0u : WS_END);
            const unsigned base = WS_STAGE + ((rank * 32 + ((ni & 1) * 2 + mi) * 8 + ((ni >> 1) & 1) * 4 + kq) * 4 + wave) * STG;
            if (total > PRE) {
#pragma unroll
                for (int v = 2; v < 5; ++v) { const unsigned w = v * 1024 + lane * 16; st128(*(const u32x4*)(pb + w), rd, (w >= PRE && w < total) ? w : OOB, base, AUX_SYS); }
                asm volatile("s_waitcnt vmcnt(0)" ::: "memory");
            }
#pragma unroll
            for (int u = 2; u >= 0; --u) {
                const unsigned o = u * 1024 + lane * 16;
                st128(d[u], rd, (o < total && o < PRE) ? o : OOB, base, AUX_SYS);
            }
        };
        const bool now = !paced || (go != 0u && (int)(go - (unsigned)rt()) <= 0);
        if (now) send();
        float mx = 0.f;
#pragma unroll
        for (int e = 0; e < 32; ++e) mx = fmaxf(mx, y[e]);
        const unsigned wmx = wave_max(asu(mx));
        if (lane == 0) {
            __hip_atomic_fetch_max((LDSP unsigned*)ctl, wmx, __ATOMIC_RELAXED, __HIP_MEMORY_SCOPE_WORKGROUP);
            if (__hip_atomic_fetch_add((LDSP unsigned*)ctl + 1, 1u, __ATOMIC_ACQ_REL, __HIP_MEMORY_SCOPE_WORKGROUP) == 3u)
                __hip_atomic_store((u64*)(ws + WS_LMAX) + b, ((u64)call << 32) | ctl[0], __ATOMIC_RELAXED, __HIP_MEMORY_SCOPE_AGENT);
        }
        if (!now) {
            while ((go = __builtin_amdgcn_readfirstlane(ctl[5])) == 0u) __builtin_amdgcn_s_sleep(1);
#pragma unroll 1
            for (int n = 0; n < 400 && (int)(go - (unsigned)rt()) > 0; ++n) __builtin_amdgcn_s_sleep(1);
            send();
        }
    } else {
        char* const my_out = p.my_out;
        const __amdgpu_buffer_rsrc_t ro = mk_rsrc(my_out, NN * NSH);
        const unsigned obase = (unsigned)(((b >> 5) * ML + ((sidx >> 2) & 1) * BM + (wave >> 1) * 128 + (sidx & 3) * 32) * NSH + (sidx >> 3) * BN + (wave & 1) * 64);
        char* const src_ws = p.ws[(b >> 5) & (NR - 1)];
        const unsigned tb = (unsigned)rt();
        unsigned call;
        {
            const int lane = threadIdx.x & 63, c16 = lane & 15, q = lane >> 4;
            if (wave == 3) {
                u32x4 t = *(const GLBP u32x4*)((GLBP const char*)(pc0 & ~255ull) + (j < CODEPF ? j : 0) * 1024 + lane * 16);
                asm volatile("" :: "v"(t));
            }
            if (wave == 0 && lane < NR) *(volatile LDSP u64*)(lds + L_CTL + 64 + lane * 8) = (u64)p.ws[lane];
            unsigned cn = 0;
            if (lane == 0) {
                cn = __hip_atomic_fetch_add((unsigned*)(ws + WS_CNT + b * 64) + wave8, 1u, __ATOMIC_RELAXED, __HIP_MEMORY_SCOPE_AGENT) + 1u;
                if (wave == 0) { ctl[0] = 0; ctl[1] = 0; ctl[2] = 0; ctl[3] = 0; ctl[5] = 0; }
                ctl[8 + wave] = 0; ctl[12 + wave] = 0;
            }
            call = __builtin_amdgcn_readfirstlane(cn);
            const unsigned xr0 = ldsb + ((wave >> 1) * 128 + c16) * 128 + ((q ^ (c16 & 7)) * 16);
            const unsigned xr1 = ldsb + ((wave >> 1) * 128 + c16) * 128 + (((4 + q) ^ (c16 & 7)) * 16);
            const unsigned wr = ldsb + 32768 + (wave & 1) * 4096 + lane * 16;
            const unsigned l16 = lane * 16, l0 = lane == 0 ? 0u : OOB, oa = ldsb + L_OWN + wave * 8192 + l16, of = ldsb + L_CTL + (8 + wave) * 4;
            unsigned pp[3], ff[3];
#pragma unroll
            for (int d = 1; d < SPLIT; ++d) {
                const int pb = (j << 3) | (np * SPLIT + (kq + d) % SPLIT);
                pp[d - 1] = WS_PART + (pb * (SPLIT - 1) + (d - 1)) * 16384 + wave * 4096;
                ff[d - 1] = WS_XFLAG + (pb * 4 + wave) * 64 + (d - 1) * 4;
            }
        asm volatile(
        "v_mov_b32 v128, 0\n\t"
        "v_mov_b32 v129, 0\n\t"
        "v_mov_b32 v130, 0\n\t"
        "v_mov_b32 v131, 0\n\t"
        "v_mov_b32 v132, 0\n\t"
        "v_mov_b32 v133, 0\n\t"
        "v_mov_b32 v134, 0\n\t"
        "v_mov_b32 v135, 0\n\t"
        "v_mov_b32 v136, 0\n\t"
        "v_mov_b32 v137, 0\n\t"
        "v_mov_b32 v138, 0\n\t"
        "v_mov_b32 v139, 0\n\t"
        "v_mov_b32 v140, 0\n\t"
        "v_mov_b32 v141, 0\n\t"
        "v_mov_b32 v142, 0\n\t"
        "v_mov_b32 v143, 0\n\t"
        "v_mov_b32 v144, 0\n\t"
        "v_mov_b32 v145, 0\n\t"
        "v_mov_b32 v146, 0\n\t"
        "v_mov_b32 v147, 0\n\t"
        "v_mov_b32 v148, 0\n\t"
        "v_mov_b32 v149, 0\n\t"
        "v_mov_b32 v150, 0\n\t"
        "v_mov_b32 v151, 0\n\t"
        "v_mov_b32 v152, 0\n\t"
        "v_mov_b32 v153, 0\n\t"
        "v_mov_b32 v154, 0\n\t"
        "v_mov_b32 v155, 0\n\t"
        "v_mov_b32 v156, 0\n\t"
        "v_mov_b32 v157, 0\n\t"
        "v_mov_b32 v158, 0\n\t"
        "v_mov_b32 v159, 0\n\t"
        "v_mov_b32 v160, 0\n\t"
        "v_mov_b32 v161, 0\n\t"
        "v_mov_b32 v162, 0\n\t"
        "v_mov_b32 v163, 0\n\t"
        "v_mov_b32 v164, 0\n\t"
        "v_mov_b32 v165, 0\n\t"
        "v_mov_b32 v166, 0\n\t"
        "v_mov_b32 v167, 0\n\t"
        "v_mov_b32 v168, 0\n\t"
        "v_mov_b32 v169, 0\n\t"
        "v_mov_b32 v170, 0\n\t"
        "v_mov_b32 v171, 0\n\t"
        "v_mov_b32 v172, 0\n\t"
        "v_mov_b32 v173, 0\n\t"
        "v_mov_b32 v174, 0\n\t"
        "v_mov_b32 v175, 0\n\t"
        "v_mov_b32 v176, 0\n\t"
        "v_mov_b32 v177, 0\n\t"
        "v_mov_b32 v178, 0\n\t"
        "v_mov_b32 v179, 0\n\t"
        "v_mov_b32 v180, 0\n\t"
        "v_mov_b32 v181, 0\n\t"
        "v_mov_b32 v182, 0\n\t"
        "v_mov_b32 v183, 0\n\t"
        "v_mov_b32 v184, 0\n\t"
        "v_mov_b32 v185, 0\n\t"
        "v_mov_b32 v186, 0\n\t"
        "v_mov_b32 v187, 0\n\t"
        "v_mov_b32 v188, 0\n\t"
        "v_mov_b32 v189, 0\n\t"
        "v_mov_b32 v190, 0\n\t"
        "v_mov_b32 v191, 0\n\t"
        "v_mov_b32 v192, 0\n\t"
        "v_mov_b32 v193, 0\n\t"
        "v_mov_b32 v194, 0\n\t"
        "v_mov_b32 v195, 0\n\t"
        "v_mov_b32 v196, 0\n\t"
        "v_mov_b32 v197, 0\n\t"
        "v_mov_b32 v198, 0\n\t"
        "v_mov_b32 v199, 0\n\t"
        "v_mov_b32 v200, 0\n\t"
        "v_mov_b32 v201, 0\n\t"
        "v_mov_b32 v202, 0\n\t"
        "v_mov_b32 v203, 0\n\t"
        "v_mov_b32 v204, 0\n\t"
        "v_mov_b32 v205, 0\n\t"
        "v_mov_b32 v206, 0\n\t"
        "v_mov_b32 v207, 0\n\t"
        "v_mov_b32 v208, 0\n\t"
        "v_mov_b32 v209, 0\n\t"
        "v_mov_b32 v210, 0\n\t"
        "v_mov_b32 v211, 0\n\t"
        "v_mov_b32 v212, 0\n\t"
        "v_mov_b32 v213, 0\n\t"
        "v_mov_b32 v214, 0\n\t"
        "v_mov_b32 v215, 0\n\t"
        "v_mov_b32 v216, 0\n\t"
        "v_mov_b32 v217, 0\n\t"
        "v_mov_b32 v218, 0\n\t"
        "v_mov_b32 v219, 0\n\t"
        "v_mov_b32 v220, 0\n\t"
        "v_mov_b32 v221, 0\n\t"
        "v_mov_b32 v222, 0\n\t"
        "v_mov_b32 v223, 0\n\t"
        "v_mov_b32 v224, 0\n\t"
        "v_mov_b32 v225, 0\n\t"
        "v_mov_b32 v226, 0\n\t"
        "v_mov_b32 v227, 0\n\t"
        "v_mov_b32 v228, 0\n\t"
        "v_mov_b32 v229, 0\n\t"
        "v_mov_b32 v230, 0\n\t"
        "v_mov_b32 v231, 0\n\t"
        "v_mov_b32 v232, 0\n\t"
        "v_mov_b32 v233, 0\n\t"
        "v_mov_b32 v234, 0\n\t"
        "v_mov_b32 v235, 0\n\t"
        "v_mov_b32 v236, 0\n\t"
        "v_mov_b32 v237, 0\n\t"
        "v_mov_b32 v238, 0\n\t"
        "v_mov_b32 v239, 0\n\t"
        "v_mov_b32 v240, 0\n\t"
        "v_mov_b32 v241, 0\n\t"
        "v_mov_b32 v242, 0\n\t"
        "v_mov_b32 v243, 0\n\t"
        "v_mov_b32 v244, 0\n\t"
        "v_mov_b32 v245, 0\n\t"
        "v_mov_b32 v246, 0\n\t"
        "v_mov_b32 v247, 0\n\t"
        "v_mov_b32 v248, 0\n\t"
        "v_mov_b32 v249, 0\n\t"
        "v_mov_b32 v250, 0\n\t"
        "v_mov_b32 v251, 0\n\t"
        "v_mov_b32 v252, 0\n\t"
        "v_mov_b32 v253, 0\n\t"
        "v_mov_b32 v254, 0\n\t"
        "v_mov_b32 v255, 0\n\t"
        "s_mov_b32 s40, 0\n\t"
        "s_mov_b32 s41, 0\n\t"
        "s_waitcnt lgkmcnt(0)\n\t"
        "CLOOP2%=:\n\t"
        "v_add_u32 v108, s41, %[xr0]\n\t"
        "v_add_u32 v109, s41, %[xr1]\n\t"
        "v_add_u32 v110, s41, %[wr]\n\t"
        "s_add_u32 s41, s41, 49152\n\t"
        "s_cmp_eq_u32 s41, 147456\n\t"
        "s_cselect_b32 s41, 0, s41\n\t"
        "s_add_u32 s40, s40, 1\n\t"
        "s_barrier\n\t"
        "ds_read_b128 v[28:31], v108 offset:0\n\t"
        "ds_read_b128 v[12:15], v110 offset:0\n\t"
        "ds_read_b128 v[16:19], v110 offset:1024\n\t"
        "ds_read_b128 v[20:23], v110 offset:2048\n\t"
        "ds_read_b128 v[24:27], v110 offset:3072\n\t"
        "ds_read_b128 v[32:35], v108 offset:2048\n\t"
        "ds_read_b128 v[36:39], v108 offset:4096\n\t"
        "ds_read_b128 v[40:43], v108 offset:6144\n\t"
        "ds_read_b128 v[44:47], v108 offset:8192\n\t"
        "ds_read_b128 v[48:51], v108 offset:10240\n\t"
        "ds_read_b128 v[52:55], v108 offset:12288\n\t"
        "ds_read_b128 v[56:59], v108 offset:14336\n\t"
        "s_waitcnt lgkmcnt(11)\n\t"
        "s_waitcnt lgkmcnt(10)\n\t"
        "v_mfma_f32_16x16x32_bf16 v[128:131], v[12:15], v[28:31], v[128:131]\n\t"
        "s_waitcnt lgkmcnt(9)\n\t"
        "v_mfma_f32_16x16x32_bf16 v[132:135], v[16:19], v[28:31], v[132:135]\n\t"
        "ds_read_b128 v[76:79], v109 offset:0\n\t"
        "s_waitcnt lgkmcnt(9)\n\t"
        "v_mfma_f32_16x16x32_bf16 v[136:139], v[20:23], v[28:31], v[136:139]\n\t"
        "s_waitcnt lgkmcnt(8)\n\t"
        "v_mfma_f32_16x16x32_bf16 v[140:143], v[24:27], v[28:31], v[140:143]\n\t"
        "ds_read_b128 v[60:63], v110 offset:8192\n\t"
        "s_waitcnt lgkmcnt(8)\n\t"
        "v_mfma_f32_16x16x32_bf16 v[156:159], v[24:27], v[32:35], v[156:159]\n\t"
        "v_mfma_f32_16x16x32_bf16 v[152:155], v[20:23], v[32:35], v[152:155]\n\t"
        "ds_read_b128 v[64:67], v110 offset:9216\n\t"
        "v_mfma_f32_16x16x32_bf16 v[148:151], v[16:19], v[32:35], v[148:151]\n\t"
        "v_mfma_f32_16x16x32_bf16 v[144:147], v[12:15], v[32:35], v[144:147]\n\t"
        "ds_read_b128 v[68:71], v110 offset:10240\n\t"
        "s_waitcnt lgkmcnt(9)\n\t"
        "v_mfma_f32_16x16x32_bf16 v[160:163], v[12:15], v[36:39], v[160:163]\n\t"
        "v_mfma_f32_16x16x32_bf16 v[164:167], v[16:19], v[36:39], v[164:167]\n\t"
        "ds_read_b128 v[72:75], v110 offset:11264\n\t"
        "v_mfma_f32_16x16x32_bf16 v[168:171], v[20:23], v[36:39], v[168:171]\n\t"
        "v_mfma_f32_16x16x32_bf16 v[172:175], v[24:27], v[36:39], v[172:175]\n\t"
        "ds_read_b128 v[80:83], v109 offset:2048\n\t"
        "s_waitcnt lgkmcnt(10)\n\t"
        "v_mfma_f32_16x16x32_bf16 v[188:191], v[24:27], v[40:43], v[188:191]\n\t"
        "v_mfma_f32_16x16x32_bf16 v[184:187], v[20:23], v[40:43], v[184:187]\n\t"
        "ds_read_b128 v[84:87], v109 offset:4096\n\t"
        "v_mfma_f32_16x16x32_bf16 v[180:183], v[16:19], v[40:43], v[180:183]\n\t"
        "v_mfma_f32_16x16x32_bf16 v[176:179], v[12:15], v[40:43], v[176:179]\n\t"
        "ds_read_b128 v[88:91], v109 offset:6144\n\t"
        "s_waitcnt lgkmcnt(11)\n\t"
        "v_mfma_f32_16x16x32_bf16 v[192:195], v[12:15], v[44:47], v[192:195]\n\t"
        "v_mfma_f32_16x16x32_bf16 v[196:199], v[16:19], v[44:47], v[196:199]\n\t"
        "ds_read_b128 v[92:95], v109 offset:8192\n\t"
        "v_mfma_f32_16x16x32_bf16 v[200:203], v[20:23], v[44:47], v[200:203]\n\t"
        "v_mfma_f32_16x16x32_bf16 v[204:207], v[24:27], v[44:47], v[204:207]\n\t"
        "ds_read_b128 v[96:99], v109 offset:10240\n\t"
        "s_waitcnt lgkmcnt(12)\n\t"
        "v_mfma_f32_16x16x32_bf16 v[220:223], v[24:27], v[48:51], v[220:223]\n\t"
        "v_mfma_f32_16x16x32_bf16 v[216:219], v[20:23], v[48:51], v[216:219]\n\t"
        "ds_read_b128 v[100:103], v109 offset:12288\n\t"
        "v_mfma_f32_16x16x32_bf16 v[212:215], v[16:19], v[48:51], v[212:215]\n\t"
        "v_mfma_f32_16x16x32_bf16 v[208:211], v[12:15], v[48:51], v[208:211]\n\t"
        "ds_read_b128 v[104:107], v109 offset:14336\n\t"
        "s_waitcnt lgkmcnt(13)\n\t"
        "v_mfma_f32_16x16x32_bf16 v[224:227], v[12:15], v[52:55], v[224:227]\n\t"
        "v_mfma_f32_16x16x32_bf16 v[228:231], v[16:19], v[52:55], v[228:231]\n\t"
        "v_mfma_f32_16x16x32_bf16 v[232:235], v[20:23], v[52:55], v[232:235]\n\t"
        "v_mfma_f32_16x16x32_bf16 v[236:239], v[24:27], v[52:55], v[236:239]\n\t"
        "s_waitcnt lgkmcnt(12)\n\t"
        "v_mfma_f32_16x16x32_bf16 v[252:255], v[24:27], v[56:59], v[252:255]\n\t"
        "v_mfma_f32_16x16x32_bf16 v[248:251], v[20:23], v[56:59], v[248:251]\n\t"
        "v_mfma_f32_16x16x32_bf16 v[244:247], v[16:19], v[56:59], v[244:247]\n\t"
        "v_mfma_f32_16x16x32_bf16 v[240:243], v[12:15], v[56:59], v[240:243]\n\t"
        "s_waitcnt lgkmcnt(11)\n\t"
        "s_waitcnt lgkmcnt(10)\n\t"
        "v_mfma_f32_16x16x32_bf16 v[128:131], v[60:63], v[76:79], v[128:131]\n\t"
        "s_waitcnt lgkmcnt(9)\n\t"
        "v_mfma_f32_16x16x32_bf16 v[132:135], v[64:67], v[76:79], v[132:135]\n\t"
        "s_waitcnt lgkmcnt(8)\n\t"
        "v_mfma_f32_16x16x32_bf16 v[136:139], v[68:71], v[76:79], v[136:139]\n\t"
        "s_waitcnt lgkmcnt(7)\n\t"
        "v_mfma_f32_16x16x32_bf16 v[140:143], v[72:75], v[76:79], v[140:143]\n\t"
        "s_waitcnt lgkmcnt(6)\n\t"
        "v_mfma_f32_16x16x32_bf16 v[156:159], v[72:75], v[80:83], v[156:159]\n\t"
        "v_mfma_f32_16x16x32_bf16 v[152:155], v[68:71], v[80:83], v[152:155]\n\t"
        "v_mfma_f32_16x16x32_bf16 v[148:151], v[64:67], v[80:83], v[148:151]\n\t"
        "v_mfma_f32_16x16x32_bf16 v[144:147], v[60:63], v[80:83], v[144:147]\n\t"
        "s_waitcnt lgkmcnt(5)\n\t"
        "v_mfma_f32_16x16x32_bf16 v[160:163], v[60:63], v[84:87], v[160:163]\n\t"
        "v_mfma_f32_16x16x32_bf16 v[164:167], v[64:67], v[84:87], v[164:167]\n\t"
        "v_mfma_f32_16x16x32_bf16 v[168:171], v[68:71], v[84:87], v[168:171]\n\t"
        "v_mfma_f32_16x16x32_bf16 v[172:175], v[72:75], v[84:87], v[172:175]\n\t"
        "s_waitcnt lgkmcnt(4)\n\t"
        "v_mfma_f32_16x16x32_bf16 v[188:191], v[72:75], v[88:91], v[188:191]\n\t"
        "v_mfma_f32_16x16x32_bf16 v[184:187], v[68:71], v[88:91], v[184:187]\n\t"
        "v_mfma_f32_16x16x32_bf16 v[180:183], v[64:67], v[88:91], v[180:183]\n\t"
        "v_mfma_f32_16x16x32_bf16 v[176:179], v[60:63], v[88:91], v[176:179]\n\t"
        "s_waitcnt lgkmcnt(3)\n\t"
        "v_mfma_f32_16x16x32_bf16 v[192:195], v[60:63], v[92:95], v[192:195]\n\t"
        "v_mfma_f32_16x16x32_bf16 v[196:199], v[64:67], v[92:95], v[196:199]\n\t"
        "v_mfma_f32_16x16x32_bf16 v[200:203], v[68:71], v[92:95], v[200:203]\n\t"
        "v_mfma_f32_16x16x32_bf16 v[204:207], v[72:75], v[92:95], v[204:207]\n\t"
        "s_waitcnt lgkmcnt(2)\n\t"
        "v_mfma_f32_16x16x32_bf16 v[220:223], v[72:75], v[96:99], v[220:223]\n\t"
        "v_mfma_f32_16x16x32_bf16 v[216:219], v[68:71], v[96:99], v[216:219]\n\t"
        "v_mfma_f32_16x16x32_bf16 v[212:215], v[64:67], v[96:99], v[212:215]\n\t"
        "v_mfma_f32_16x16x32_bf16 v[208:211], v[60:63], v[96:99], v[208:211]\n\t"
        "s_waitcnt lgkmcnt(1)\n\t"
        "v_mfma_f32_16x16x32_bf16 v[224:227], v[60:63], v[100:103], v[224:227]\n\t"
        "v_mfma_f32_16x16x32_bf16 v[228:231], v[64:67], v[100:103], v[228:231]\n\t"
        "v_mfma_f32_16x16x32_bf16 v[232:235], v[68:71], v[100:103], v[232:235]\n\t"
        "v_mfma_f32_16x16x32_bf16 v[236:239], v[72:75], v[100:103], v[236:239]\n\t"
        "s_waitcnt lgkmcnt(0)\n\t"
        "v_mfma_f32_16x16x32_bf16 v[252:255], v[72:75], v[104:107], v[252:255]\n\t"
        "v_mfma_f32_16x16x32_bf16 v[248:251], v[68:71], v[104:107], v[248:251]\n\t"
        "v_mfma_f32_16x16x32_bf16 v[244:247], v[64:67], v[104:107], v[244:247]\n\t"
        "v_mfma_f32_16x16x32_bf16 v[240:243], v[60:63], v[104:107], v[240:243]\n\t"
        "s_cmp_lt_u32 s40, 15\n\t"
        "s_cbranch_scc1 CLOOP2%=\n\t"
        "v_add_u32 v108, s41, %[xr0]\n\t"
        "v_add_u32 v109, s41, %[xr1]\n\t"
        "v_add_u32 v110, s41, %[wr]\n\t"
        "s_add_u32 s41, s41, 49152\n\t"
        "s_cmp_eq_u32 s41, 147456\n\t"
        "s_cselect_b32 s41, 0, s41\n\t"
        "s_add_u32 s40, s40, 1\n\t"
        "s_barrier\n\t"
        "ds_read_b128 v[36:39], v108 offset:4096\n\t"
        "ds_read_b128 v[12:15], v110 offset:0\n\t"
        "ds_read_b128 v[16:19], v110 offset:1024\n\t"
        "ds_read_b128 v[20:23], v110 offset:2048\n\t"
        "ds_read_b128 v[24:27], v110 offset:3072\n\t"
        "ds_read_b128 v[40:43], v108 offset:6144\n\t"
        "ds_read_b128 v[84:87], v109 offset:4096\n\t"
        "ds_read_b128 v[60:63], v110 offset:8192\n\t"
        "ds_read_b128 v[64:67], v110 offset:9216\n\t"
        "ds_read_b128 v[68:71], v110 offset:10240\n\t"
        "ds_read_b128 v[72:75], v110 offset:11264\n\t"
        "ds_read_b128 v[88:91], v109 offset:6144\n\t"
        "s_waitcnt lgkmcnt(11)\n\t"
        "s_waitcnt lgkmcnt(10)\n\t"
        "v_mfma_f32_16x16x32_bf16 v[160:163], v[12:15], v[36:39], v[160:163]\n\t"
        "ds_read_b128 v[44:47], v108 offset:8192\n\t"
        "s_waitcnt lgkmcnt(10)\n\t"
        "v_mfma_f32_16x16x32_bf16 v[164:167], v[16:19], v[36:39], v[164:167]\n\t"
        "ds_read_b128 v[48:51], v108 offset:10240\n\t"
        "s_waitcnt lgkmcnt(10)\n\t"
        "v_mfma_f32_16x16x32_bf16 v[168:171], v[20:23], v[36:39], v[168:171]\n\t"
        "ds_read_b128 v[92:95], v109 offset:8192\n\t"
        "s_waitcnt lgkmcnt(10)\n\t"
        "v_mfma_f32_16x16x32_bf16 v[172:175], v[24:27], v[36:39], v[172:175]\n\t"
        "ds_read_b128 v[96:99], v109 offset:10240\n\t"
        "s_waitcnt lgkmcnt(10)\n\t"
        "v_mfma_f32_16x16x32_bf16 v[188:191], v[24:27], v[40:43], v[188:191]\n\t"
        "ds_read_b128 v[52:55], v108 offset:12288\n\t"
        "v_mfma_f32_16x16x32_bf16 v[184:187], v[20:23], v[40:43], v[184:187]\n\t"
        "ds_read_b128 v[56:59], v108 offset:14336\n\t"
        "v_mfma_f32_16x16x32_bf16 v[180:183], v[16:19], v[40:43], v[180:183]\n\t"
        "ds_read_b128 v[100:103], v109 offset:12288\n\t"
        "v_mfma_f32_16x16x32_bf16 v[176:179], v[12:15], v[40:43], v[176:179]\n\t"
        "ds_read_b128 v[104:107], v109 offset:14336\n\t"
        "s_waitcnt lgkmcnt(13)\n\t"
        "s_waitcnt lgkmcnt(12)\n\t"
        "v_mfma_f32_16x16x32_bf16 v[160:163], v[60:63], v[84:87], v[160:163]\n\t"
        "ds_read_b128 v[28:31], v108 offset:0\n\t"
        "s_waitcnt lgkmcnt(12)\n\t"
        "v_mfma_f32_16x16x32_bf16 v[164:167], v[64:67], v[84:87], v[164:167]\n\t"
        "ds_read_b128 v[32:35], v108 offset:2048\n\t"
        "s_waitcnt lgkmcnt(12)\n\t"
        "v_mfma_f32_16x16x32_bf16 v[168:171], v[68:71], v[84:87], v[168:171]\n\t"
        "ds_read_b128 v[76:79], v109 offset:0\n\t"
        "s_waitcnt lgkmcnt(12)\n\t"
        "v_mfma_f32_16x16x32_bf16 v[172:175], v[72:75], v[84:87], v[172:175]\n\t"
        "ds_read_b128 v[80:83], v109 offset:2048\n\t"
        "s_waitcnt lgkmcnt(12)\n\t"
        "v_mfma_f32_16x16x32_bf16 v[188:191], v[72:75], v[88:91], v[188:191]\n\t"
        "v_mfma_f32_16x16x32_bf16 v[184:187], v[68:71], v[88:91], v[184:187]\n\t"
        "v_mfma_f32_16x16x32_bf16 v[180:183], v[64:67], v[88:91], v[180:183]\n\t"
        "v_mfma_f32_16x16x32_bf16 v[176:179], v[60:63], v[88:91], v[176:179]\n\t"
        "s_waitcnt lgkmcnt(11)\n\t"
        "v_mfma_f32_16x16x32_bf16 v[192:195], v[12:15], v[44:47], v[192:195]\n\t"
        "v_mfma_f32_16x16x32_bf16 v[196:199], v[16:19], v[44:47], v[196:199]\n\t"
        "v_mfma_f32_16x16x32_bf16 v[200:203], v[20:23], v[44:47], v[200:203]\n\t"
        "v_mfma_f32_16x16x32_bf16 v[204:207], v[24:27], v[44:47], v[204:207]\n\t"
        "s_waitcnt lgkmcnt(10)\n\t"
        "v_mfma_f32_16x16x32_bf16 v[220:223], v[24:27], v[48:51], v[220:223]\n\t"
        "v_mfma_f32_16x16x32_bf16 v[216:219], v[20:23], v[48:51], v[216:219]\n\t"
        "v_mfma_f32_16x16x32_bf16 v[212:215], v[16:19], v[48:51], v[212:215]\n\t"
        "v_mfma_f32_16x16x32_bf16 v[208:211], v[12:15], v[48:51], v[208:211]\n\t"
        "s_waitcnt lgkmcnt(9)\n\t"
        "v_mfma_f32_16x16x32_bf16 v[192:195], v[60:63], v[92:95], v[192:195]\n\t"
        "v_cvt_pk_bf16_f32 v112, v160, v161\n\t"
        "v_cvt_pk_bf16_f32 v113, v162, v163\n\t"
        "v_cvt_pk_bf16_f32 v114, v164, v165\n\t"
        "v_mfma_f32_16x16x32_bf16 v[196:199], v[64:67], v[92:95], v[196:199]\n\t"
        "v_cvt_pk_bf16_f32 v115, v166, v167\n\t"
        "buffer_store_dwordx4 v[112:115], %[l16], %[rs], %[p1] offen offset:0 sc1\n\t"
        "v_cvt_pk_bf16_f32 v116, v168, v169\n\t"
        "v_mfma_f32_16x16x32_bf16 v[200:203], v[68:71], v[92:95], v[200:203]\n\t"
        "v_cvt_pk_bf16_f32 v117, v170, v171\n\t"
        "v_cvt_pk_bf16_f32 v118, v172, v173\n\t"
        "v_cvt_pk_bf16_f32 v119, v174, v175\n\t"
        "v_mfma_f32_16x16x32_bf16 v[204:207], v[72:75], v[92:95], v[204:207]\n\t"
        "buffer_store_dwordx4 v[116:119], %[l16], %[rs], %[p1] offen offset:1024 sc1\n\t"
        "v_cvt_pk_bf16_f32 v120, v176, v177\n\t"
        "v_cvt_pk_bf16_f32 v121, v178, v179\n\t"
        "s_waitcnt lgkmcnt(8)\n\t"
        "v_mfma_f32_16x16x32_bf16 v[220:223], v[72:75], v[96:99], v[220:223]\n\t"
        "v_cvt_pk_bf16_f32 v122, v180, v181\n\t"
        "v_cvt_pk_bf16_f32 v123, v182, v183\n\t"
        "buffer_store_dwordx4 v[120:123], %[l16], %[rs], %[p1] offen offset:2048 sc1\n\t"
        "v_mfma_f32_16x16x32_bf16 v[216:219], v[68:71], v[96:99], v[216:219]\n\t"
        "v_cvt_pk_bf16_f32 v124, v184, v185\n\t"
        "v_cvt_pk_bf16_f32 v125, v186, v187\n\t"
        "v_cvt_pk_bf16_f32 v126, v188, v189\n\t"
        "v_mfma_f32_16x16x32_bf16 v[212:215], v[64:67], v[96:99], v[212:215]\n\t"
        "v_cvt_pk_bf16_f32 v127, v190, v191\n\t"
        "buffer_store_dwordx4 v[124:127], %[l16], %[rs], %[p1] offen offset:3072 sc1\n\t"
        "v_mfma_f32_16x16x32_bf16 v[208:211], v[60:63], v[96:99], v[208:211]\n\t"
        "s_waitcnt lgkmcnt(7)\n\t"
        "v_mfma_f32_16x16x32_bf16 v[224:227], v[12:15], v[52:55], v[224:227]\n\t"
        "v_mfma_f32_16x16x32_bf16 v[228:231], v[16:19], v[52:55], v[228:231]\n\t"
        "v_mfma_f32_16x16x32_bf16 v[232:235], v[20:23], v[52:55], v[232:235]\n\t"
        "v_mfma_f32_16x16x32_bf16 v[236:239], v[24:27], v[52:55], v[236:239]\n\t"
        "s_waitcnt lgkmcnt(6)\n\t"
        "v_mfma_f32_16x16x32_bf16 v[252:255], v[24:27], v[56:59], v[252:255]\n\t"
        "v_mfma_f32_16x16x32_bf16 v[248:251], v[20:23], v[56:59], v[248:251]\n\t"
        "v_mfma_f32_16x16x32_bf16 v[244:247], v[16:19], v[56:59], v[244:247]\n\t"
        "v_mfma_f32_16x16x32_bf16 v[240:243], v[12:15], v[56:59], v[240:243]\n\t"
        "s_waitcnt lgkmcnt(5)\n\t"
        "v_mfma_f32_16x16x32_bf16 v[224:227], v[60:63], v[100:103], v[224:227]\n\t"
        "v_cvt_pk_bf16_f32 v36, v192, v193\n\t"
        "v_cvt_pk_bf16_f32 v37, v194, v195\n\t"
        "v_cvt_pk_bf16_f32 v38, v196, v197\n\t"
        "v_mfma_f32_16x16x32_bf16 v[228:231], v[64:67], v[100:103], v[228:231]\n\t"
        "v_cvt_pk_bf16_f32 v39, v198, v199\n\t"
        "buffer_store_dwordx4 v[36:39], %[l16], %[rs], %[p2] offen offset:0 sc1\n\t"
        "v_cvt_pk_bf16_f32 v40, v200, v201\n\t"
        "v_mfma_f32_16x16x32_bf16 v[232:235], v[68:71], v[100:103], v[232:235]\n\t"
        "v_cvt_pk_bf16_f32 v41, v202, v203\n\t"
        "v_cvt_pk_bf16_f32 v42, v204, v205\n\t"
        "v_cvt_pk_bf16_f32 v43, v206, v207\n\t"
        "v_mfma_f32_16x16x32_bf16 v[236:239], v[72:75], v[100:103], v[236:239]\n\t"
        "buffer_store_dwordx4 v[40:43], %[l16], %[rs], %[p2] offen offset:1024 sc1\n\t"
        "v_cvt_pk_bf16_f32 v44, v208, v209\n\t"
        "v_cvt_pk_bf16_f32 v45, v210, v211\n\t"
        "s_waitcnt lgkmcnt(4)\n\t"
        "v_mfma_f32_16x16x32_bf16 v[252:255], v[72:75], v[104:107], v[252:255]\n\t"
        "v_cvt_pk_bf16_f32 v46, v212, v213\n\t"
        "v_cvt_pk_bf16_f32 v47, v214, v215\n\t"
        "buffer_store_dwordx4 v[44:47], %[l16], %[rs], %[p2] offen offset:2048 sc1\n\t"
        "v_mfma_f32_16x16x32_bf16 v[248:251], v[68:71], v[104:107], v[248:251]\n\t"
        "v_cvt_pk_bf16_f32 v48, v216, v217\n\t"
        "v_cvt_pk_bf16_f32 v49, v218, v219\n\t"
        "v_cvt_pk_bf16_f32 v50, v220, v221\n\t"
        "v_mfma_f32_16x16x32_bf16 v[244:247], v[64:67], v[104:107], v[244:247]\n\t"
        "v_cvt_pk_bf16_f32 v51, v222, v223\n\t"
        "buffer_store_dwordx4 v[48:51], %[l16], %[rs], %[p2] offen offset:3072 sc1\n\t"
        "v_mfma_f32_16x16x32_bf16 v[240:243], v[60:63], v[104:107], v[240:243]\n\t"
        "s_waitcnt lgkmcnt(3)\n\t"
        "v_mfma_f32_16x16x32_bf16 v[128:131], v[12:15], v[28:31], v[128:131]\n\t"
        "v_mfma_f32_16x16x32_bf16 v[132:135], v[16:19], v[28:31], v[132:135]\n\t"
        "v_mfma_f32_16x16x32_bf16 v[136:139], v[20:23], v[28:31], v[136:139]\n\t"
        "v_mfma_f32_16x16x32_bf16 v[140:143], v[24:27], v[28:31], v[140:143]\n\t"
        "s_waitcnt lgkmcnt(2)\n\t"
        "v_mfma_f32_16x16x32_bf16 v[156:159], v[24:27], v[32:35], v[156:159]\n\t"
        "v_mfma_f32_16x16x32_bf16 v[152:155], v[20:23], v[32:35], v[152:155]\n\t"
        "v_mfma_f32_16x16x32_bf16 v[148:151], v[16:19], v[32:35], v[148:151]\n\t"
        "v_mfma_f32_16x16x32_bf16 v[144:147], v[12:15], v[32:35], v[144:147]\n\t"
        "s_waitcnt lgkmcnt(1)\n\t"
        "v_mfma_f32_16x16x32_bf16 v[128:131], v[60:63], v[76:79], v[128:131]\n\t"
        "v_cvt_pk_bf16_f32 v84, v224, v225\n\t"
        "v_cvt_pk_bf16_f32 v85, v226, v227\n\t"
        "v_cvt_pk_bf16_f32 v86, v228, v229\n\t"
        "v_mfma_f32_16x16x32_bf16 v[132:135], v[64:67], v[76:79], v[132:135]\n\t"
        "v_cvt_pk_bf16_f32 v87, v230, v231\n\t"
        "buffer_store_dwordx4 v[84:87], %[l16], %[rs], %[p3] offen offset:0 sc1\n\t"
        "v_cvt_pk_bf16_f32 v88, v232, v233\n\t"
        "v_mfma_f32_16x16x32_bf16 v[136:139], v[68:71], v[76:79], v[136:139]\n\t"
        "v_cvt_pk_bf16_f32 v89, v234, v235\n\t"
        "v_cvt_pk_bf16_f32 v90, v236, v237\n\t"
        "v_cvt_pk_bf16_f32 v91, v238, v239\n\t"
        "v_mfma_f32_16x16x32_bf16 v[140:143], v[72:75], v[76:79], v[140:143]\n\t"
        "buffer_store_dwordx4 v[88:91], %[l16], %[rs], %[p3] offen offset:1024 sc1\n\t"
        "v_cvt_pk_bf16_f32 v92, v240, v241\n\t"
        "v_cvt_pk_bf16_f32 v93, v242, v243\n\t"
        "s_waitcnt lgkmcnt(0)\n\t"
        "v_mfma_f32_16x16x32_bf16 v[156:159], v[72:75], v[80:83], v[156:159]\n\t"
        "v_cvt_pk_bf16_f32 v94, v244, v245\n\t"
        "v_cvt_pk_bf16_f32 v95, v246, v247\n\t"
        "buffer_store_dwordx4 v[92:95], %[l16], %[rs], %[p3] offen offset:2048 sc1\n\t"
        "v_mfma_f32_16x16x32_bf16 v[152:155], v[68:71], v[80:83], v[152:155]\n\t"
        "v_cvt_pk_bf16_f32 v96, v248, v249\n\t"
        "v_cvt_pk_bf16_f32 v97, v250, v251\n\t"
        "v_cvt_pk_bf16_f32 v98, v252, v253\n\t"
        "v_mfma_f32_16x16x32_bf16 v[148:151], v[64:67], v[80:83], v[148:151]\n\t"
        "v_cvt_pk_bf16_f32 v99, v254, v255\n\t"
        "buffer_store_dwordx4 v[96:99], %[l16], %[rs], %[p3] offen offset:3072 sc1\n\t"
        "v_mfma_f32_16x16x32_bf16 v[144:147], v[60:63], v[80:83], v[144:147]\n\t"
        "s_nop 7\n\t"
        "ds_write_b128 %[oa], v[128:131] offset:0\n\t"
        "ds_write_b128 %[oa], v[132:135] offset:1024\n\t"
        "ds_write_b128 %[oa], v[136:139] offset:2048\n\t"
        "ds_write_b128 %[oa], v[140:143] offset:3072\n\t"
        "ds_write_b128 %[oa], v[144:147] offset:4096\n\t"
        "ds_write_b128 %[oa], v[148:151] offset:5120\n\t"
        "ds_write_b128 %[oa], v[152:155] offset:6144\n\t"
        "ds_write_b128 %[oa], v[156:159] offset:7168\n\t"
        "v_mov_b32 v110, 1\n\t"
        "v_mov_b32 v108, %[call]\n\t"
        "s_waitcnt lgkmcnt(0)\n\t"
        "ds_write_b32 %[of], v110\n\t"
        "s_waitcnt vmcnt(8)\n\t"
        "buffer_store_dword v108, %[l0], %[rs], %[f1] offen sc1\n\t"
        "s_waitcnt vmcnt(5)\n\t"
        "buffer_store_dword v108, %[l0], %[rs], %[f2] offen sc1\n\t"
        "s_waitcnt vmcnt(2)\n\t"
        "buffer_store_dword v108, %[l0], %[rs], %[f3] offen sc1\n\t"
        "s_waitcnt lgkmcnt(0)\n\t"
        :
        : [xr0] "v"(xr0), [xr1] "v"(xr1), [wr] "v"(wr), [l16] "v"(l16), [l0] "v"(l0), [oa] "v"(oa), [of] "v"(of), [rs] "s"(rws), [call] "s"(call),
          [p1] "s"(pp[0]), [p2] "s"(pp[1]), [p3] "s"(pp[2]), [f1] "s"(ff[0]), [f2] "s"(ff[1]), [f3] "s"(ff[2])
        : "memory", "scc", "v12", "v13", "v14", "v15", "v16", "v17", "v18", "v19", "v20", "v21", "v22", "v23", "v24", "v25", "v26", "v27", "v28", "v29", "v30", "v31", "v32", "v33", "v34", "v35", "v36", "v37", "v38", "v39", "v40", "v41", "v42", "v43", "v44", "v45", "v46", "v47", "v48", "v49", "v50", "v51", "v52", "v53", "v54", "v55", "v56", "v57", "v58", "v59", "v60", "v61", "v62", "v63", "v64", "v65", "v66", "v67", "v68", "v69", "v70", "v71", "v72", "v73", "v74", "v75", "v76", "v77", "v78", "v79", "v80", "v81", "v82", "v83", "v84", "v85", "v86", "v87", "v88", "v89", "v90", "v91", "v92", "v93", "v94", "v95", "v96", "v97", "v98", "v99", "v100", "v101", "v102", "v103", "v104", "v105", "v106", "v107", "v108", "v109", "v110", "v111", "v112", "v113", "v114", "v115", "v116", "v117", "v118", "v119", "v120", "v121", "v122", "v123", "v124", "v125", "v126", "v127", "v128", "v129", "v130", "v131", "v132", "v133", "v134", "v135", "v136", "v137", "v138", "v139", "v140", "v141", "v142", "v143", "v144", "v145", "v146", "v147", "v148", "v149", "v150", "v151", "v152", "v153", "v154", "v155", "v156", "v157", "v158", "v159", "v160", "v161", "v162", "v163", "v164", "v165", "v166", "v167", "v168", "v169", "v170", "v171", "v172", "v173", "v174", "v175", "v176", "v177", "v178", "v179", "v180", "v181", "v182", "v183", "v184", "v185", "v186", "v187", "v188", "v189", "v190", "v191", "v192", "v193", "v194", "v195", "v196", "v197", "v198", "v199", "v200", "v201", "v202", "v203", "v204", "v205", "v206", "v207", "v208", "v209", "v210", "v211", "v212", "v213", "v214", "v215", "v216", "v217", "v218", "v219", "v220", "v221", "v222", "v223", "v224", "v225", "v226", "v227", "v228", "v229", "v230", "v231", "v232", "v233", "v234", "v235", "v236", "v237", "v238", "v239", "v240", "v241", "v242", "v243", "v244", "v245", "v246", "v247", "v248", "v249", "v250", "v251", "v252", "v253", "v254", "v255", "s40", "s41");

        }
        const int lane = lane_id(), c16 = lane & 15, q = lane >> 4;
        const unsigned par = call & 1u;
        const unsigned t0 = (unsigned)rt();
        while (__builtin_amdgcn_readfirstlane(ctl[12 + wave]) == 0u) __builtin_amdgcn_s_sleep(1);
        asm volatile("" ::: "memory");
        {
            const int sni = rank * 4 + (sidx >> 3);
            const int sxr = (sni >> 2) * 2 + (sni & 1), snp = ((sni >> 1) ^ (sni >> 2)) & 1;
            const int sb = ((sxr * 2 + ((sidx >> 2) & 1)) << 3) | (snp * 4 + (sidx & 3));
            if (lane == 0) __hip_atomic_store((unsigned*)(src_ws + WS_XFLAG + (sb * 4 + wave) * 64) + 4, call, __ATOMIC_RELAXED, __HIP_MEMORY_SCOPE_SYSTEM);
        }
        auto timed_out = [&]() { return (unsigned)__builtin_amdgcn_readfirstlane(__builtin_amdgcn_raw_buffer_load_b32(rws, 0, WS_TMO, AUX_AGENT)) == call; };
        auto give_up = [&]() {
            const unsigned dt = (unsigned)rt() - t0;
            if (dt > 5000u && (dt > DEADLINE_TICKS || timed_out())) {
                if (lane == 0) __hip_atomic_store(tmo, call, __ATOMIC_RELAXED, __HIP_MEMORY_SCOPE_AGENT);
                return true;
            }
            return false;
        };
        if (wave == 1 && b == 0) {
            GLBP char* const lane_ws = (GLBP char*)*(volatile LDSP u64*)(lds + L_CTL + 64 + (lane & (NR - 1)) * 8);
            while (true) {
                const u32x4 g0 = __builtin_amdgcn_raw_buffer_load_b128(rws, lane * 16, WS_LMAX, AUX_AGENT);
                const u32x4 g1 = __builtin_amdgcn_raw_buffer_load_b128(rws, lane * 16 + 1024, WS_LMAX, AUX_AGENT);
                if (__all(g0[1] == call && g0[3] == call && g1[1] == call && g1[3] == call)) {
                    const unsigned gm = wave_max(max(max(g0[0], g0[2]), max(g1[0], g1[2])));
                    __hip_atomic_store((GLBP u64*)(lane_ws + WS_GMAX + (par * 8 + (lane >> 3)) * 128) + rank, ((u64)call << 32) | gm, __ATOMIC_RELAXED, __HIP_MEMORY_SCOPE_SYSTEM);
                    break;
                }
                if (give_up()) break;
                asm volatile("" ::: "memory");
            }
        }
        bool bad = false, have_max = wave != 0;
        unsigned mask = 0, ib = 0;
        const unsigned poff = WS_GMAX + (par * 8 + (b & 7)) * 128 + (lane & 7) * 8;
        auto try_max = [&](const u32x2 g) {
            if (!have_max && __all(lane >= NR || g[1] == call)) {
                const float scale = asf(row_max(lane < NR ? g[0] : 0u)) / 448.f;
                ib = __builtin_amdgcn_readfirstlane(asu(scale > 0.f ? 1.f / scale : 0.f));
                if (lane == 0) { ctl[2] = ib; __builtin_amdgcn_fence(__ATOMIC_RELEASE, "workgroup"); ctl[3] = 1u; }
                have_max = true;
            }
        };
        const unsigned pl = ldsb + L_PK + wave * PK, fv = sbase + 256 + lane * 16;
        auto fetch3 = [&]() {
            unsigned keepm;
            asm volatile("s_mov_b32 %0, m0\n\ts_mov_b32 m0, %3\n\ts_nop 0\n\tbuffer_load_dwordx4 %1, %2, 0 offen sc0 sc1 lds\n\t"
                         "buffer_load_dwordx4 %1, %2, 0 offen offset:1024 sc0 sc1 lds\n\tbuffer_load_dwordx4 %1, %2, 0 offen offset:2048 sc0 sc1 lds\n\ts_mov_b32 m0, %0"
                         : "=&s"(keepm) : "v"(fv), "s"(rws), "s"(pl) : "memory");
        };
        auto fetch1 = [&]() {
            unsigned keepm;
            asm volatile("s_mov_b32 %0, m0\n\ts_mov_b32 m0, %3\n\ts_nop 0\n\tbuffer_load_dwordx4 %1, %2, 0 offen offset:3072 sc0 sc1 lds\n\ts_mov_b32 m0, %0"
                         : "=&s"(keepm) : "v"(fv), "s"(rws), "s"(pl) : "memory");
        };
        while (true) {
            mask = __builtin_amdgcn_raw_buffer_load_b32(rws, lane * 4, sbase, AUX_SYS);
            u32x2 g = {0, 0};
            if (!have_max) g = __builtin_amdgcn_raw_buffer_load_b64(rws, poff, 0, AUX_SYS);
            asm volatile("" : "+v"(g), "+v"(mask));
            const bool got = __all((int)mask >= 0);
            if (got) fetch3();
            try_max(g);
            if (got) break;
            if (give_up()) { bad = true; mask = 0; break; }
            __builtin_amdgcn_s_sleep(5);
            asm volatile("" ::: "memory");
        }
        unsigned total, ad[32], keep[8];
        ad[0] = pl + wave_scan(__builtin_popcount(mask) + 1, total) * 2u;
        if (total > 1536u) fetch1();
#pragma unroll
        for (int e = 0; e < 31; ++e)
            asm("v_bfe_u32 %0, %1, %2, 1\n\tv_lshl_add_u32 %0, %0, 1, %3" : "=&v"(ad[e + 1]) : "v"(mask), "i"((e >> 1) + (e & 1) * 16), "v"(ad[e]));
#pragma unroll
        for (int k = 0; k < 8; ++k) {
            unsigned t = __umul24(((mask | 0x80000000u) >> (8 * (k >> 2) + ((k >> 1) & 1) + 16 * (k & 1))) & 0x55u, 0x41041u) & 0x01010101u, u = t << 7;
            asm volatile("" : "+v"(u));
            keep[k] = u - t;
            asm volatile("" : "+v"(keep[k]));
        }
        float y[32];
#pragma unroll
        for (int e = 0; e < 32; ++e) y[e] = 0.f;
        while (true) {
            asm volatile("s_waitcnt vmcnt(0)" ::: "memory");
#pragma unroll
            for (int e = 0; e < 32; ++e) asm volatile("ds_read_u16_d16_hi %0, %1" : "+v"(y[e]) : "v"(ad[e]) : "memory");
            asm volatile("s_waitcnt lgkmcnt(0)" : "+v"(y[0]), "+v"(y[1]), "+v"(y[2]), "+v"(y[3]), "+v"(y[4]), "+v"(y[5]), "+v"(y[6]), "+v"(y[7]),
                         "+v"(y[8]), "+v"(y[9]), "+v"(y[10]), "+v"(y[11]), "+v"(y[12]), "+v"(y[13]), "+v"(y[14]), "+v"(y[15]) :: "memory");
            asm volatile("" : "+v"(y[16]), "+v"(y[17]), "+v"(y[18]), "+v"(y[19]), "+v"(y[20]), "+v"(y[21]), "+v"(y[22]), "+v"(y[23]),
                         "+v"(y[24]), "+v"(y[25]), "+v"(y[26]), "+v"(y[27]), "+v"(y[28]), "+v"(y[29]), "+v"(y[30]), "+v"(y[31]) :: "memory");
            unsigned any = 0;
#pragma unroll
            for (int e = 0; e < 32; ++e) any |= asu(y[e]);
            if (bad || __all((int)any >= 0)) break;
            fetch3(); fetch1();
            if (give_up()) { bad = true; break; }
        }
        if (wave == 0) {
            while (!have_max) {
                const u32x2 g = __builtin_amdgcn_raw_buffer_load_b64(rws, poff, 0, AUX_SYS);
                try_max(g);
                if (!have_max && give_up()) {
                    ib = 0xffffffffu;
                    if (lane == 0) { ctl[2] = ib; __builtin_amdgcn_fence(__ATOMIC_RELEASE, "workgroup"); ctl[3] = 1u; }
                    break;
                }
                asm volatile("" ::: "memory");
            }
        } else {
            while (__builtin_amdgcn_readfirstlane(ctl[3]) == 0u) __builtin_amdgcn_s_sleep(1);
            __builtin_amdgcn_fence(__ATOMIC_ACQUIRE, "workgroup");
            ib = __builtin_amdgcn_readfirstlane(ctl[2]);
        }
        const unsigned nan = (bad || ib == 0xffffffffu || ((unsigned)rt() - tb > 5000u && timed_out())) ? 0x7f7f7f7fu : 0u;
        st128(quant16(y, keep, asf(ib), nan), ro, c16 * NSH + q * 16, obase, 0);
        st128(quant16(y + 16, keep + 4, asf(ib), nan), ro, (16 + c16) * NSH + q * 16, obase, 0);
    }
}

extern "C" void kernel_launch(int rank, int nranks, ncclComm_t comm,
                              hipStream_t stream,
                              void* const* d_in, const int* in_sizes, int n_in,
                              void* d_out, int out_size,
                              const int* in_dtypes, int out_dtype,
                              void* const* d_out_all,
                              void* const* d_ws_all, size_t ws_bytes) {
    if (nranks != NR || n_in != 2 || in_sizes[0] != ML * KK || in_sizes[1] != KK * NN || out_size != NN * NSH ||
        ws_bytes < WS_END || (unsigned)rank >= (unsigned)NR) {
        fprintf(stderr, "kernel_launch: unexpected problem shape\n");
        return;
    }
    static bool attr_set[NR] = {};
    if (!attr_set[rank]) {
        HIP_CHECK(hipFuncSetAttribute((const void*)fused_gemm_a2a, hipFuncAttributeMaxDynamicSharedMemorySize, LDS_BYTES));
        attr_set[rank] = true;
    }
    Params p;
    p.x = (const float*)d_in[0];
    p.w = (const float*)d_in[1];
    for (int r = 0; r < NR; ++r) p.ws[r] = (char*)d_ws_all[r];
    p.my_out = (char*)d_out; p.my_ws = p.ws[rank];
    p.rank = rank; p.pad = 0;
    fused_gemm_a2a<<<NWG, 512, LDS_BYTES, stream>>>(p);
    HIP_CHECK(hipGetLastError());
}
```
